```python
import numpy as np

kernel_name = "gpu_gemm_a2a_m4096_k4096_n4096_f32_qe4m3"


ABSMAX_RTOL = 0.09

COMPUTE_DTYPE = "float32"
EPILOGUE = "quant_e4m3"
IN_QUANT = None
OUT_QUANT = "e4m3"
IN_DTYPES = {"x": "float32", "w_mat": "float32"}
WORLD_SIZE = 8

def setup_inputs():

    x = np.random.randn(4096, 4096).astype(np.float32)
    w_mat = (np.random.randn(4096, 4096) / np.sqrt(4096)).astype(np.float32)
    return {"x": x, "w_mat": w_mat}

def _snap_fp8(x, mant_bits, max_abs, min_normal):
    a = np.abs(x.astype(np.float64))
    _m, _e = np.frexp(a)
    q = np.where(a >= min_normal, np.ldexp(1.0, _e - 1 - mant_bits),
                 min_normal * 2.0 ** -mant_bits)
    snapped = np.minimum(np.round(a / q) * q, max_abs)
    return (np.sign(x) * snapped).astype(np.float32)
def _quant_dequant(y, amax):
    scale = amax / 448.0
    q = _snap_fp8(y / scale, 3, 448.0, 2.0 ** -6)
    return (q * scale).astype(np.float32)
def _epilogue(y):
    return _quant_dequant(y, np.abs(y).max())

def reference(x, w_mat):
    y = (x @ w_mat).astype(np.float32)
    return _epilogue(y)
```

```cpp
#include <hip/hip_runtime.h>
#include <rccl/rccl.h>
#include <cstdio>
#include <cstdint>

#define HIP_CHECK(expr) do { hipError_t _e = (expr); if (_e != hipSuccess) { \
    fprintf(stderr, "HIP error %s at %s:%d\n", hipGetErrorString(_e), __FILE__, __LINE__); \
    return; } } while (0)

namespace {

typedef __attribute__((ext_vector_type(4))) float f32x4;
typedef __attribute__((ext_vector_type(4))) unsigned u32x4;
typedef __attribute__((ext_vector_type(8))) __bf16 bf16x8;

constexpr int NR = 8;
constexpr int MR = 512;
constexpr int KD = 4096;
constexpr int ND = 4096;
constexpr int NS = 512;
constexpr int BM = 64, BN = 128;
constexpr int NWG = (MR / BM) * (ND / BN);
constexpr unsigned long long DEADLINE_TICKS = 50000000ull;

constexpr size_t WS_TICKET = 0;
constexpr size_t WS_CTR1   = 128;
constexpr size_t WS_CTR2   = 256;
constexpr size_t WS_TMO    = 384;
constexpr size_t WS_AMAX   = 1024;
constexpr size_t WS_DONE   = 2048;
constexpr size_t WS_WGMAX  = 4096;
constexpr size_t WS_END    = WS_WGMAX + NWG * 8;

__device__ __forceinline__ bf16x8 pack8(float a0, float a1, float a2, float a3,
                                        float a4, float a5, float a6, float a7) {
    bf16x8 r;
    r[0] = (__bf16)a0; r[1] = (__bf16)a1; r[2] = (__bf16)a2; r[3] = (__bf16)a3;
    r[4] = (__bf16)a4; r[5] = (__bf16)a5; r[6] = (__bf16)a6; r[7] = (__bf16)a7;
    return r;
}

struct Stage {
    f32x4 w[2][8];
    f32x4 x[4][2];
};

__device__ __forceinline__ void load_stage(Stage& s, const float* __restrict__ wp,
                                           const float* __restrict__ xp, int kb) {
    const float* w = wp + (size_t)kb * 32 * ND;
    const float* x = xp + kb * 32;
#pragma unroll
    for (int i = 0; i < 8; ++i)
#pragma unroll
        for (int g = 0; g < 2; ++g)
            s.w[g][i] = *reinterpret_cast<const f32x4*>(w + (size_t)((i & 3) + (i >> 2) * 16) * ND + g * 64);
#pragma unroll
    for (int rt = 0; rt < 4; ++rt)
#pragma unroll
        for (int h2 = 0; h2 < 2; ++h2)
            s.x[rt][h2] = *reinterpret_cast<const f32x4*>(x + (size_t)rt * 16 * KD + h2 * 16);
}

__device__ __forceinline__ void compute_stage(const Stage& s, f32x4 (&acc)[2][4][4]) {
    bf16x8 xf[4];
#pragma unroll
    for (int rt = 0; rt < 4; ++rt)
        xf[rt] = pack8(s.x[rt][0][0], s.x[rt][0][1], s.x[rt][0][2], s.x[rt][0][3],
                       s.x[rt][1][0], s.x[rt][1][1], s.x[rt][1][2], s.x[rt][1][3]);
#pragma unroll
    for (int g = 0; g < 2; ++g)
#pragma unroll
        for (int j = 0; j < 4; ++j) {
            bf16x8 wf = pack8(s.w[g][0][j], s.w[g][1][j], s.w[g][2][j], s.w[g][3][j],
                              s.w[g][4][j], s.w[g][5][j], s.w[g][6][j], s.w[g][7][j]);
#pragma unroll
            for (int rt = 0; rt < 4; ++rt)
                acc[g][j][rt] = __builtin_amdgcn_mfma_f32_16x16x32_bf16(wf, xf[rt], acc[g][j][rt], 0, 0, 0);
        }
}

__device__ __forceinline__ unsigned pack_fp8x4(float a, float b, float c, float d) {
    int r = 0;
    r = __builtin_amdgcn_cvt_pk_fp8_f32(a, b, r, false);
    r = __builtin_amdgcn_cvt_pk_fp8_f32(c, d, r, true);
    return (unsigned)r;
}

__global__ void __launch_bounds__(256, 1)
fused_gemm_a2a(const float* __restrict__ X, const float* __restrict__ W,
               void* const* __restrict__ out_all, void* const* __restrict__ ws_all, int rank) {
    __shared__ __attribute__((aligned(16))) char smem[4 * 4 * 8 * 1024 + 64];
    f32x4* red = reinterpret_cast<f32x4*>(smem);
    unsigned* sbc = reinterpret_cast<unsigned*>(smem + 4 * 4 * 8 * 1024);

    const int tid = threadIdx.x, lane = tid & 63;
    const int wave = __builtin_amdgcn_readfirstlane(tid >> 6);
    const int c = lane & 15, h = lane >> 4;
    const int b = blockIdx.x;
    const int xcd = b & 7, idx = b >> 3;
    const int mt = idx & 7, nt = xcd * 4 + (idx >> 3);
    const int m0 = mt * BM, n0 = nt * BN;

    char* ws = reinterpret_cast<char*>(ws_all[rank]);
    unsigned call = 0;
    if (tid == 0) {
        unsigned t = __hip_atomic_fetch_add(reinterpret_cast<unsigned*>(ws + WS_TICKET), 1u,
                                            __ATOMIC_RELAXED, __HIP_MEMORY_SCOPE_AGENT);
        call = t / NWG + 1;
    }

    f32x4 acc[2][4][4];
#pragma unroll
    for (int g = 0; g < 2; ++g)
#pragma unroll
        for (int j = 0; j < 4; ++j)
#pragma unroll
            for (int rt = 0; rt < 4; ++rt) acc[g][j][rt] = f32x4{0.f, 0.f, 0.f, 0.f};

    const float* wp = W + (size_t)(4 * h) * ND + n0 + 4 * c;
    const float* xp = X + (size_t)(m0 + c) * KD + 4 * h;

    Stage sa, sb;
    load_stage(sa, wp, xp, wave);
    constexpr int NIT = KD / 32 / 4;
#pragma unroll 1
    for (int it = 0; it < NIT; it += 2) {
        load_stage(sb, wp, xp, wave + 4 * (it + 1));
        compute_stage(sa, acc);
        if (it + 2 < NIT) load_stage(sa, wp, xp, wave + 4 * (it + 2));
        compute_stage(sb, acc);
    }

#pragma unroll
    for (int rt = 0; rt < 4; ++rt) {
        if (rt != wave) {
#pragma unroll
            for (int g = 0; g < 2; ++g)
#pragma unroll
                for (int j = 0; j < 4; ++j)
                    red[((wave * 4 + rt) * 8 + g * 4 + j) * 64 + lane] = acc[g][j][rt];
        }
    }
    __syncthreads();
    f32x4 fin[2][4];
#pragma unroll
    for (int g = 0; g < 2; ++g)
#pragma unroll
        for (int j = 0; j < 4; ++j) {
            f32x4 v = f32x4{0.f, 0.f, 0.f, 0.f};
#pragma unroll
            for (int rt = 0; rt < 4; ++rt) if (rt == wave) v = acc[g][j][rt];
            fin[g][j] = v;
        }
#pragma unroll
    for (int sw = 0; sw < 4; ++sw) {
        if (sw != wave) {
#pragma unroll
            for (int g = 0; g < 2; ++g)
#pragma unroll
                for (int j = 0; j < 4; ++j)
                    fin[g][j] += red[((sw * 4 + wave) * 8 + g * 4 + j) * 64 + lane];
        }
    }

    float am = 0.f;
#pragma unroll
    for (int g = 0; g < 2; ++g)
#pragma unroll
        for (int j = 0; j < 4; ++j)
#pragma unroll
            for (int r = 0; r < 4; ++r) am = fmaxf(am, fabsf(fin[g][j][r]));
#pragma unroll
    for (int off = 32; off > 0; off >>= 1) am = fmaxf(am, __shfl_xor(am, off));
    if (lane == 0) sbc[wave] = __float_as_uint(am);
    __syncthreads();

    unsigned* tmo = reinterpret_cast<unsigned*>(ws + WS_TMO);
    if (wave == 0) {
        call = __shfl(call, 0);
        unsigned last = 0;
        if (lane == 0) {
            unsigned a = max(max(sbc[0], sbc[1]), max(sbc[2], sbc[3]));
            __hip_atomic_store(reinterpret_cast<unsigned long long*>(ws + WS_WGMAX + b * 8),
                               ((unsigned long long)call << 32) | a, __ATOMIC_RELAXED, __HIP_MEMORY_SCOPE_AGENT);
            unsigned t = __hip_atomic_fetch_add(reinterpret_cast<unsigned*>(ws + WS_CTR1), 1u,
                                                __ATOMIC_RELAXED, __HIP_MEMORY_SCOPE_AGENT);
            last = (t == call * NWG - 1);
        }
        last = __shfl(last, 0);
        unsigned long long t0 = __builtin_amdgcn_s_memrealtime();
        if (last) {
            unsigned a = 0;
            bool ok = false;
            while (!ok) {
                ok = true; a = 0;
#pragma unroll
                for (int q = 0; q < NWG / 64; ++q) {
                    unsigned long long v = __hip_atomic_load(
                        reinterpret_cast<unsigned long long*>(ws + WS_WGMAX + (q * 64 + lane) * 8),
                        __ATOMIC_RELAXED, __HIP_MEMORY_SCOPE_AGENT);
                    ok = ok && ((unsigned)(v >> 32) == call);
                    a = max(a, (unsigned)v);
                }
                ok = __all(ok);
                if (!ok) {
                    if (__builtin_amdgcn_s_memrealtime() - t0 > DEADLINE_TICKS) {
                        if (lane == 0) __hip_atomic_store(tmo, call, __ATOMIC_RELAXED, __HIP_MEMORY_SCOPE_AGENT);
                        break;
                    }
                    __builtin_amdgcn_s_sleep(1);
                }
            }
#pragma unroll
            for (int off = 32; off > 0; off >>= 1) a = max(a, (unsigned)__shfl_xor((int)a, off));
            if (ok && lane < NR) {
                char* pws = reinterpret_cast<char*>(ws_all[lane]);
                __hip_atomic_store(reinterpret_cast<unsigned long long*>(pws + WS_AMAX + rank * 128),
                                   ((unsigned long long)call << 32) | a, __ATOMIC_RELAXED, __HIP_MEMORY_SCOPE_SYSTEM);
            }
        }
        unsigned ga = 0;
        bool good = false;
        while (!good) {
            bool ok = true;
            if (lane < NR) {
                unsigned long long v = __hip_atomic_load(
                    reinterpret_cast<unsigned long long*>(ws + WS_AMAX + lane * 128),
                    __ATOMIC_RELAXED, __HIP_MEMORY_SCOPE_SYSTEM);
                ok = ((unsigned)(v >> 32) == call);
                ga = (unsigned)v;
            }
            good = __all(ok);
            if (!good) {
                unsigned tv = __hip_atomic_load(tmo, __ATOMIC_RELAXED, __HIP_MEMORY_SCOPE_AGENT);
                if (tv == call || __builtin_amdgcn_s_memrealtime() - t0 > DEADLINE_TICKS) {
                    if (lane == 0) __hip_atomic_store(tmo, call, __ATOMIC_RELAXED, __HIP_MEMORY_SCOPE_AGENT);
                    break;
                }
                __builtin_amdgcn_s_sleep(1);
            }
        }
#pragma unroll
        for (int off = 4; off > 0; off >>= 1) ga = max(ga, (unsigned)__shfl_xor((int)ga, off));
        if (lane == 0) { sbc[4] = ga; sbc[5] = good ? 1u : 0u; sbc[6] = call; }
    }
    __syncthreads();
    const float gmax = __uint_as_float(sbc[4]);
    const bool good = sbc[5] != 0;
    call = sbc[6];

    const float inv = gmax > 0.f ? 448.0f / gmax : 0.f;
    const unsigned dst = (unsigned)(n0 / NS);
    if (dst < (unsigned)NR) {
        char* op = reinterpret_cast<char*>(out_all[dst]) +
                   (size_t)(rank * MR + m0 + 16 * wave + c) * NS + (n0 % NS) + 16 * h;
#pragma unroll
        for (int g = 0; g < 2; ++g) {
            u32x4 q;
#pragma unroll
            for (int r = 0; r < 4; ++r) {
                float v0 = __builtin_amdgcn_fmed3f(fin[g][0][r] * inv, -448.f, 448.f);
                float v1 = __builtin_amdgcn_fmed3f(fin[g][1][r] * inv, -448.f, 448.f);
                float v2 = __builtin_amdgcn_fmed3f(fin[g][2][r] * inv, -448.f, 448.f);
                float v3 = __builtin_amdgcn_fmed3f(fin[g][3][r] * inv, -448.f, 448.f);
                q[r] = good ? pack_fp8x4(v0, v1, v2, v3) : 0x7f7f7f7fu;
            }
            *reinterpret_cast<u32x4*>(op + 64 * g) = q;
        }
    }
    asm volatile("s_waitcnt vmcnt(0)" ::: "memory");
    __syncthreads();
    if (wave == 0) {
        unsigned last = 0;
        if (lane == 0) {
            __builtin_amdgcn_fence(__ATOMIC_RELEASE, "");
            asm volatile("s_waitcnt vmcnt(0)" ::: "memory");
            unsigned t = __hip_atomic_fetch_add(reinterpret_cast<unsigned*>(ws + WS_CTR2), 1u,
                                                __ATOMIC_RELEASE, __HIP_MEMORY_SCOPE_AGENT);
            last = (t == call * NWG - 1);
        }
        last = __shfl(last, 0);
        if (last) {
            __builtin_amdgcn_fence(__ATOMIC_ACQUIRE, "agent");
            __builtin_amdgcn_fence(__ATOMIC_RELEASE, "");
            asm volatile("s_waitcnt vmcnt(0)" ::: "memory");
            if (lane < NR) {
                char* pws = reinterpret_cast<char*>(ws_all[lane]);
                __hip_atomic_store(reinterpret_cast<unsigned*>(pws + WS_DONE + rank * 128), call,
                                   __ATOMIC_RELAXED, __HIP_MEMORY_SCOPE_SYSTEM);
            }
            unsigned long long t0 = __builtin_amdgcn_s_memrealtime();
            bool fine = false;
            while (!fine) {
                bool ok = true;
                if (lane < NR) {
                    unsigned v = __hip_atomic_load(reinterpret_cast<unsigned*>(ws + WS_DONE + lane * 128),
                                                   __ATOMIC_RELAXED, __HIP_MEMORY_SCOPE_SYSTEM);
                    ok = (v >= call);
                }
                fine = __all(ok);
                if (!fine) {
                    unsigned tv = __hip_atomic_load(tmo, __ATOMIC_RELAXED, __HIP_MEMORY_SCOPE_AGENT);
                    if (tv == call || __builtin_amdgcn_s_memrealtime() - t0 > DEADLINE_TICKS) {
                        if (lane == 0) {
                            __hip_atomic_store(tmo, call, __ATOMIC_RELAXED, __HIP_MEMORY_SCOPE_AGENT);
                            reinterpret_cast<unsigned char*>(out_all[rank])[(size_t)rank * MR * NS] = 0x7f;
                        }
                        break;
                    }
                    __builtin_amdgcn_s_sleep(1);
                }
            }
            __builtin_amdgcn_fence(__ATOMIC_ACQUIRE, "");
        }
    }
}

}

extern "C" void kernel_launch(int rank, int nranks, ncclComm_t comm,
                              hipStream_t stream,
                              void* const* d_in, const int* in_sizes, int n_in,
                              void* d_out, int out_size,
                              const int* in_dtypes, int out_dtype,
                              void* const* d_out_all,
                              void* const* d_ws_all, size_t ws_bytes) {
    (void)comm; (void)in_sizes; (void)n_in; (void)d_out; (void)out_size; (void)in_dtypes; (void)out_dtype;
    if (nranks != NR || ws_bytes < WS_END) {
        fprintf(stderr, "unsupported configuration\n");
        return;
    }
    hipLaunchKernelGGL(fused_gemm_a2a, dim3(NWG), dim3(256), 0, stream,
                       (const float*)d_in[0], (const float*)d_in[1], d_out_all, d_ws_all, rank);
    HIP_CHECK(hipGetLastError());
}
```

```cpp
#include <hip/hip_runtime.h>
#include <rccl/rccl.h>
#include <cstdio>
#include <cstdint>

#define HIP_CHECK(expr) do { hipError_t _e = (expr); if (_e != hipSuccess) { \
    fprintf(stderr, "HIP error %s at %s:%d\n", hipGetErrorString(_e), __FILE__, __LINE__); \
    return; } } while (0)

namespace {

typedef __attribute__((ext_vector_type(4))) float f32x4;
typedef __attribute__((ext_vector_type(4))) unsigned u32x4;
typedef __attribute__((ext_vector_type(2))) unsigned u32x2;
typedef __attribute__((ext_vector_type(8))) __bf16 bf16x8;
typedef __attribute__((ext_vector_type(4))) __bf16 bf16x4;
typedef __attribute__((ext_vector_type(2))) __bf16 bf16x2;

constexpr int NR = 8;
constexpr int MR = 512;
constexpr int KD = 4096;
constexpr int ND = 4096;
constexpr int NS = 512;
constexpr int BM = 256, BN = 128, BK = 32, SK = 4, KS = KD / SK, NT = KS / BK;
constexpr int NWG = (MR / BM) * (ND / BN) * SK;
constexpr int NTHR = 512;
constexpr int PD = 2;
constexpr unsigned long long DEADLINE_TICKS = 50000000ull;

constexpr size_t WS_TICKET = 0;
constexpr size_t WS_CTR1   = 128;
constexpr size_t WS_CTR2   = 256;
constexpr size_t WS_TMO    = 384;
constexpr size_t WS_AMAX   = 1024;
constexpr size_t WS_DONE   = 2048;
constexpr size_t WS_WGMAX  = 4096;
constexpr size_t WS_GCTR   = 8192;
constexpr size_t WS_SCR    = 1 << 20;
constexpr size_t SCR_BYTES = (size_t)64 * 16 * 8 * 2048;
constexpr size_t WS_END    = WS_SCR + SCR_BYTES;

__device__ __forceinline__ unsigned cvt2(float a, float b) {
    bf16x2 r; r[0] = (__bf16)a; r[1] = (__bf16)b; return __builtin_bit_cast(unsigned, r);
}
__device__ __forceinline__ f32x4 ldb(__amdgpu_buffer_rsrc_t r, int voff, int soff) {
    return __builtin_bit_cast(f32x4, __builtin_amdgcn_raw_buffer_load_b128(r, voff, soff, 0));
}
__device__ __forceinline__ unsigned pack_fp8x4(float a, float b, float c, float d) {
    int r = 0;
    r = __builtin_amdgcn_cvt_pk_fp8_f32(a, b, r, false);
    r = __builtin_amdgcn_cvt_pk_fp8_f32(c, d, r, true);
    return (unsigned)r;
}
#define SB0 __builtin_amdgcn_sched_barrier(0)

__global__ void __launch_bounds__(NTHR, 2)
fused_gemm_a2a(const float* __restrict__ X, const float* __restrict__ W,
               void* const* __restrict__ out_all, void* const* __restrict__ ws_all, int rank) {
    constexpr int XB = BM * 64, WB = BK * 256;
    __shared__ __attribute__((aligned(16))) char smem[2 * (XB + WB) + 64];
    unsigned* sbc = reinterpret_cast<unsigned*>(smem + 2 * (XB + WB));

    const int tid = threadIdx.x, lane = tid & 63;
    const int wave = __builtin_amdgcn_readfirstlane(tid >> 6);
    const int c = lane & 15, h = lane >> 4;
    const int wm = wave >> 1, wn = wave & 1;
    const int b = blockIdx.x, xcd = b & 7, idx = b >> 3;
    const int sl = xcd & 3, mt = idx & 1, nt = (xcd >> 2) * 16 + (idx >> 1);
    const int m0 = mt * BM, n0 = nt * BN, k0 = sl * KS;
    const int grp = mt * 32 + nt;

    char* ws = reinterpret_cast<char*>(ws_all[rank]);
    unsigned call = 0;
    if (tid == 0) {
        unsigned t = __hip_atomic_fetch_add(reinterpret_cast<unsigned*>(ws + WS_TICKET), 1u,
                                            __ATOMIC_RELAXED, __HIP_MEMORY_SCOPE_AGENT);
        call = t / NWG + 1;
    }

    __amdgpu_buffer_rsrc_t rx = __builtin_amdgcn_make_buffer_rsrc((void*)X, 0, MR * KD * 4, 0x00020000);
    __amdgpu_buffer_rsrc_t rw = __builtin_amdgcn_make_buffer_rsrc((void*)W, 0, KD * ND * 4, 0x00020000);
    const int xrow = tid >> 2, xj = tid & 3;
    const int xv = ((m0 + xrow) * KD + k0 + 4 * xj) * 4;
    const int wk = tid >> 5, wq = tid & 31;
    const int wv = ((k0 + wk) * ND + n0 + 4 * wq) * 4;
    const int fsw = (0x1320 >> (((xrow >> 2) & 3) * 4)) & 3;
    const int xw_off = xrow * 64 + ((xj ^ fsw) * 16);
    const int ww_off = wk * 256 + ((wq ^ ((wk & 3) | (((wk >> 2) & 1) << 4))) * 8);
    const int fr = (0x1320 >> (((c >> 2) & 3) * 4)) & 3;
    const int xr_off = (64 * wm + c) * 64 + ((h ^ fr) * 16);
    const int q4 = c >> 2, p4 = c & 3;
    int wr_off[4];
#pragma unroll
    for (int t = 0; t < 4; ++t)
        wr_off[t] = (4 * h + q4) * 256 + ((((16 * wn) ^ ((h & 1) << 4)) + 4 * p4 + (t ^ q4)) * 8);

    f32x4 acc[4][4];
#pragma unroll
    for (int t = 0; t < 4; ++t)
#pragma unroll
        for (int u = 0; u < 4; ++u) acc[t][u] = f32x4{0.f, 0.f, 0.f, 0.f};

    f32x4 xr[PD][4], wr[PD][2];
    bf16x8 xf[2][4], wf[2][4];
    auto issue = [&](int st, int set) {
        xr[set][0] = ldb(rx, xv, st * 128);
        xr[set][1] = ldb(rx, xv + 64, st * 128);
        xr[set][2] = ldb(rx, xv, st * 128 + 128 * KD * 4);
        xr[set][3] = ldb(rx, xv + 64, st * 128 + 128 * KD * 4);
        wr[set][0] = ldb(rw, wv, st * 32 * ND * 4);
        wr[set][1] = ldb(rw, wv, (st * 32 + 16) * ND * 4);
    };
    auto stash = [&](int set, int buf) {
        char* xb = smem + buf * (XB + WB); char* wb = xb + XB;
#pragma unroll
        for (int ps = 0; ps < 2; ++ps) {
            u32x4 v;
            v[0] = cvt2(xr[set][2 * ps][0], xr[set][2 * ps][1]);
            v[1] = cvt2(xr[set][2 * ps][2], xr[set][2 * ps][3]);
            v[2] = cvt2(xr[set][2 * ps + 1][0], xr[set][2 * ps + 1][1]);
            v[3] = cvt2(xr[set][2 * ps + 1][2], xr[set][2 * ps + 1][3]);
            *reinterpret_cast<u32x4*>(xb + xw_off + ps * 128 * 64) = v;
            u32x2 w2;
            w2[0] = cvt2(wr[set][ps][0], wr[set][ps][1]);
            w2[1] = cvt2(wr[set][ps][2], wr[set][ps][3]);
            *reinterpret_cast<u32x2*>(wb + ww_off + ps * 16 * 256) = w2;
        }
    };
    auto frags = [&](int buf, int fs) {
        char* xb = smem + buf * (XB + WB); char* wb = xb + XB;
#pragma unroll
        for (int u = 0; u < 4; ++u) xf[fs][u] = *reinterpret_cast<const bf16x8*>(xb + xr_off + u * 1024);
#pragma unroll
        for (int tt = 0; tt < 4; ++tt) {
            bf16x4 lo = __builtin_amdgcn_ds_read_tr16_b64_v4bf16((__attribute__((address_space(3))) bf16x4*)(wb + wr_off[tt]));
            bf16x4 hi = __builtin_amdgcn_ds_read_tr16_b64_v4bf16((__attribute__((address_space(3))) bf16x4*)(wb + wr_off[tt] + 4096));
            wf[fs][tt] = __builtin_shufflevector(lo, hi, 0, 1, 2, 3, 4, 5, 6, 7);
        }
    };
    auto mma = [&](int fs) {
#pragma unroll
        for (int tt = 0; tt < 4; ++tt)
#pragma unroll
            for (int u = 0; u < 4; ++u)
                acc[tt][u] = __builtin_amdgcn_mfma_f32_16x16x32_bf16(wf[fs][tt], xf[fs][u], acc[tt][u], 0, 0, 0);
    };
#pragma unroll
    for (int d = 0; d < PD; ++d) issue(d, d);
    SB0;
    stash(0, 0); issue(PD, 0);
    SB0;
    __syncthreads();
    SB0;
    frags(0, 0);
    stash(1 % PD, 1); issue(PD + 1, 1 % PD);
    SB0;
    __syncthreads();
    SB0;
#pragma unroll
    for (int t = 0; t < NT; ++t) {
        if (t + 1 < NT) frags((t + 1) & 1, (t + 1) & 1);
        if (t + 2 < NT) {
            stash((t + 2) % PD, t & 1);
            if (t + 2 + PD < NT) issue(t + 2 + PD, (t + 2) % PD);
        }
        mma(t & 1);
        SB0;
        if (t + 1 < NT) __syncthreads();
        SB0;
    }

    unsigned* tmo = reinterpret_cast<unsigned*>(ws + WS_TMO);
    __amdgpu_buffer_rsrc_t rs = __builtin_amdgcn_make_buffer_rsrc((void*)(ws + WS_SCR), 0, (int)SCR_BYTES, 0x00020000);
#pragma unroll
    for (int u = 0; u < 4; ++u) if (u != sl) {
#pragma unroll
        for (int hf = 0; hf < 2; ++hf) {
            u32x4 v;
            v[0] = cvt2(acc[2 * hf][u][0], acc[2 * hf][u][1]);
            v[1] = cvt2(acc[2 * hf][u][2], acc[2 * hf][u][3]);
            v[2] = cvt2(acc[2 * hf + 1][u][0], acc[2 * hf + 1][u][1]);
            v[3] = cvt2(acc[2 * hf + 1][u][2], acc[2 * hf + 1][u][3]);
            __builtin_amdgcn_raw_buffer_store_b128(v, rs, lane * 16, ((((grp * 4 + u) * 4 + sl) * 8 + wave) * 2 + hf) * 1024, 16);
        }
    }
    asm volatile("s_waitcnt vmcnt(0)" ::: "memory");
    __syncthreads();
    const unsigned long long t0 = __builtin_amdgcn_s_memrealtime();
    if (tid == 0) {
        unsigned* ctr = reinterpret_cast<unsigned*>(ws + WS_GCTR + grp * 128);
        __hip_atomic_fetch_add(ctr, 1u, __ATOMIC_RELAXED, __HIP_MEMORY_SCOPE_AGENT);
        while (__hip_atomic_load(ctr, __ATOMIC_RELAXED, __HIP_MEMORY_SCOPE_AGENT) < call * SK) {
            if (__builtin_amdgcn_s_memrealtime() - t0 > DEADLINE_TICKS) {
                __hip_atomic_store(tmo, call, __ATOMIC_RELAXED, __HIP_MEMORY_SCOPE_AGENT);
                break;
            }
            __builtin_amdgcn_s_sleep(1);
        }
    }
    __syncthreads();
    asm volatile("" ::: "memory");
    f32x4 fin[4];
#pragma unroll
    for (int tt = 0; tt < 4; ++tt) {
        f32x4 v = f32x4{0.f, 0.f, 0.f, 0.f};
#pragma unroll
        for (int u = 0; u < 4; ++u) if (u == sl) v = acc[tt][u];
        fin[tt] = v;
    }
    {
        u32x4 pv[3][2];
#pragma unroll
        for (int i = 0; i < 3; ++i) {
            int src = (sl + 1 + i) & 3;
#pragma unroll
            for (int hf = 0; hf < 2; ++hf)
                pv[i][hf] = __builtin_amdgcn_raw_buffer_load_b128(rs, lane * 16, ((((grp * 4 + sl) * 4 + src) * 8 + wave) * 2 + hf) * 1024, 16);
        }
#pragma unroll
        for (int i = 0; i < 3; ++i)
#pragma unroll
            for (int hf = 0; hf < 2; ++hf)
#pragma unroll
                for (int e = 0; e < 4; ++e) {
                    fin[2 * hf + (e >> 1)][(e & 1) * 2]     += __uint_as_float(pv[i][hf][e] << 16);
                    fin[2 * hf + (e >> 1)][(e & 1) * 2 + 1] += __uint_as_float(pv[i][hf][e] & 0xffff0000u);
                }
    }

    float am = 0.f;
#pragma unroll
    for (int tt = 0; tt < 4; ++tt)
#pragma unroll
        for (int r = 0; r < 4; ++r) am = fmaxf(am, fabsf(fin[tt][r]));
#pragma unroll
    for (int off = 32; off > 0; off >>= 1) am = fmaxf(am, __shfl_xor(am, off));
    if (lane == 0) sbc[wave] = __float_as_uint(am);
    __syncthreads();

    if (wave == 0) {
        call = __shfl(call, 0);
        unsigned last = 0;
        if (lane == 0) {
            unsigned a = max(max(max(sbc[0], sbc[1]), max(sbc[2], sbc[3])),
                             max(max(sbc[4], sbc[5]), max(sbc[6], sbc[7])));
            __hip_atomic_store(reinterpret_cast<unsigned long long*>(ws + WS_WGMAX + b * 8),
                               ((unsigned long long)call << 32) | a, __ATOMIC_RELAXED, __HIP_MEMORY_SCOPE_AGENT);
            unsigned t = __hip_atomic_fetch_add(reinterpret_cast<unsigned*>(ws + WS_CTR1), 1u,
                                                __ATOMIC_RELAXED, __HIP_MEMORY_SCOPE_AGENT);
            last = (t == call * NWG - 1);
        }
        last = __shfl(last, 0);
        if (last) {
            unsigned a = 0;
            bool ok = false;
            while (!ok) {
                ok = true; a = 0;
#pragma unroll
                for (int q = 0; q < NWG / 64; ++q) {
                    unsigned long long v = __hip_atomic_load(
                        reinterpret_cast<unsigned long long*>(ws + WS_WGMAX + (q * 64 + lane) * 8),
                        __ATOMIC_RELAXED, __HIP_MEMORY_SCOPE_AGENT);
                    ok = ok && ((unsigned)(v >> 32) == call);
                    a = max(a, (unsigned)v);
                }
                ok = __all(ok);
                if (!ok) {
                    if (__builtin_amdgcn_s_memrealtime() - t0 > DEADLINE_TICKS) {
                        if (lane == 0) __hip_atomic_store(tmo, call, __ATOMIC_RELAXED, __HIP_MEMORY_SCOPE_AGENT);
                        break;
                    }
                    __builtin_amdgcn_s_sleep(1);
                }
            }
#pragma unroll
            for (int off = 32; off > 0; off >>= 1) a = max(a, (unsigned)__shfl_xor((int)a, off));
            if (ok && lane < NR) {
                char* pws = reinterpret_cast<char*>(ws_all[lane]);
                __hip_atomic_store(reinterpret_cast<unsigned long long*>(pws + WS_AMAX + rank * 128),
                                   ((unsigned long long)call << 32) | a, __ATOMIC_RELAXED, __HIP_MEMORY_SCOPE_SYSTEM);
            }
        }
        unsigned ga = 0;
        bool good = false;
        while (!good) {
            bool ok = true;
            if (lane < NR) {
                unsigned long long v = __hip_atomic_load(
                    reinterpret_cast<unsigned long long*>(ws + WS_AMAX + lane * 128),
                    __ATOMIC_RELAXED, __HIP_MEMORY_SCOPE_SYSTEM);
                ok = ((unsigned)(v >> 32) == call);
                ga = (unsigned)v;
            }
            good = __all(ok);
            if (!good) {
                unsigned tv = __hip_atomic_load(tmo, __ATOMIC_RELAXED, __HIP_MEMORY_SCOPE_AGENT);
                if (tv == call || __builtin_amdgcn_s_memrealtime() - t0 > DEADLINE_TICKS) {
                    if (lane == 0) __hip_atomic_store(tmo, call, __ATOMIC_RELAXED, __HIP_MEMORY_SCOPE_AGENT);
                    break;
                }
                __builtin_amdgcn_s_sleep(1);
            }
        }
        if (good) {
            unsigned tv = __hip_atomic_load(tmo, __ATOMIC_RELAXED, __HIP_MEMORY_SCOPE_AGENT);
            if (tv == call) good = false;
        }
#pragma unroll
        for (int off = 4; off > 0; off >>= 1) ga = max(ga, (unsigned)__shfl_xor((int)ga, off));
        if (lane == 0) { sbc[8] = ga; sbc[9] = good ? 1u : 0u; sbc[10] = call; }
    }
    __syncthreads();
    const float gmax = __uint_as_float(sbc[8]);
    const bool good = sbc[9] != 0;
    call = sbc[10];

    const float inv = gmax > 0.f ? 448.0f / gmax : 0.f;
    const unsigned dst = (unsigned)(n0 / NS);
    if (dst < (unsigned)NR) {
        char* op = reinterpret_cast<char*>(out_all[dst]) +
                   (size_t)(rank * MR + m0 + 64 * wm + 16 * sl + c) * NS + (n0 % NS) + 64 * wn + 16 * h;
        u32x4 q;
#pragma unroll
        for (int tt = 0; tt < 4; ++tt) {
            float v0 = __builtin_amdgcn_fmed3f(fin[tt][0] * inv, -448.f, 448.f);
            float v1 = __builtin_amdgcn_fmed3f(fin[tt][1] * inv, -448.f, 448.f);
            float v2 = __builtin_amdgcn_fmed3f(fin[tt][2] * inv, -448.f, 448.f);
            float v3 = __builtin_amdgcn_fmed3f(fin[tt][3] * inv, -448.f, 448.f);
            q[tt] = good ? pack_fp8x4(v0, v1, v2, v3) : 0x7f7f7f7fu;
        }
        *reinterpret_cast<u32x4*>(op) = q;
    }
    asm volatile("s_waitcnt vmcnt(0)" ::: "memory");
    __syncthreads();
    if (wave == 0) {
        unsigned last = 0;
        if (lane == 0) {
            __builtin_amdgcn_fence(__ATOMIC_RELEASE, "");
            asm volatile("s_waitcnt vmcnt(0)" ::: "memory");
            unsigned t = __hip_atomic_fetch_add(reinterpret_cast<unsigned*>(ws + WS_CTR2), 1u,
                                                __ATOMIC_RELEASE, __HIP_MEMORY_SCOPE_AGENT);
            last = (t == call * NWG - 1);
        }
        last = __shfl(last, 0);
        if (last) {
            __builtin_amdgcn_fence(__ATOMIC_ACQUIRE, "agent");
            __builtin_amdgcn_fence(__ATOMIC_RELEASE, "");
            asm volatile("s_waitcnt vmcnt(0)" ::: "memory");
            if (lane < NR) {
                char* pws = reinterpret_cast<char*>(ws_all[lane]);
                __hip_atomic_store(reinterpret_cast<unsigned*>(pws + WS_DONE + rank * 128), call,
                                   __ATOMIC_RELAXED, __HIP_MEMORY_SCOPE_SYSTEM);
            }
            unsigned long long t1 = __builtin_amdgcn_s_memrealtime();
            bool fine = false;
            while (!fine) {
                bool ok = true;
                if (lane < NR) {
                    unsigned v = __hip_atomic_load(reinterpret_cast<unsigned*>(ws + WS_DONE + lane * 128),
                                                   __ATOMIC_RELAXED, __HIP_MEMORY_SCOPE_SYSTEM);
                    ok = (v >= call);
                }
                fine = __all(ok);
                if (!fine) {
                    unsigned tv = __hip_atomic_load(tmo, __ATOMIC_RELAXED, __HIP_MEMORY_SCOPE_AGENT);
                    if (tv == call || __builtin_amdgcn_s_memrealtime() - t1 > DEADLINE_TICKS) {
                        if (lane == 0) {
                            __hip_atomic_store(tmo, call, __ATOMIC_RELAXED, __HIP_MEMORY_SCOPE_AGENT);
                            reinterpret_cast<unsigned char*>(out_all[rank])[(size_t)rank * MR * NS] = 0x7f;
                        }
                        break;
                    }
                    __builtin_amdgcn_s_sleep(1);
                }
            }
            __builtin_amdgcn_fence(__ATOMIC_ACQUIRE, "");
        }
    }
}

}

extern "C" void kernel_launch(int rank, int nranks, ncclComm_t comm,
                              hipStream_t stream,
                              void* const* d_in, const int* in_sizes, int n_in,
                              void* d_out, int out_size,
                              const int* in_dtypes, int out_dtype,
                              void* const* d_out_all,
                              void* const* d_ws_all, size_t ws_bytes) {
    (void)comm; (void)in_sizes; (void)n_in; (void)d_out; (void)out_size; (void)in_dtypes; (void)out_dtype;
    if (nranks != NR || ws_bytes < WS_END) {
        fprintf(stderr, "unsupported configuration (ws_bytes %zu)\n", ws_bytes);
        return;
    }
    hipLaunchKernelGGL(fused_gemm_a2a, dim3(NWG), dim3(NTHR), 0, stream,
                       (const float*)d_in[0], (const float*)d_in[1], d_out_all, d_ws_all, rank);
    HIP_CHECK(hipGetLastError());
}
```

```cpp
#include <hip/hip_runtime.h>
#include <rccl/rccl.h>
#include <cstdio>
#include <cstdint>

#define HIP_CHECK(expr) do { hipError_t _e = (expr); if (_e != hipSuccess) { \
    fprintf(stderr, "HIP error %s at %s:%d\n", hipGetErrorString(_e), __FILE__, __LINE__); \
    return; } } while (0)

namespace {

typedef __attribute__((ext_vector_type(4))) float f32x4;
typedef __attribute__((ext_vector_type(4))) unsigned u32x4;
typedef __attribute__((ext_vector_type(2))) unsigned u32x2;
typedef __attribute__((ext_vector_type(8))) __bf16 bf16x8;
typedef __attribute__((ext_vector_type(4))) __bf16 bf16x4;
typedef __attribute__((ext_vector_type(2))) __bf16 bf16x2;

constexpr int NR = 8;
constexpr int MR = 512;
constexpr int KD = 4096;
constexpr int ND = 4096;
constexpr int NS = 512;
constexpr int BM = 256, BN = 128, BK = 32, SK = 4, KS = KD / SK, NT = KS / BK;
constexpr int NWG = (MR / BM) * (ND / BN) * SK;
constexpr int NTHR = 512;
constexpr int AGG_WG = 0, WAIT_WG = 0;
constexpr int PD = 2;
constexpr unsigned long long DEADLINE_TICKS = 50000000ull;

constexpr size_t WS_TICKET = 0;
constexpr size_t WS_CTR1   = 128;
constexpr size_t WS_CTR2   = 256;
constexpr size_t WS_TMO    = 384;
constexpr size_t WS_AMAX   = 1024;
constexpr size_t WS_DONE   = 2048;
constexpr size_t WS_WGMAX  = 4096;
constexpr size_t WS_GCTR   = 8192;
constexpr size_t WS_SCR    = 1 << 20;
constexpr size_t SCR_BYTES = (size_t)64 * 16 * 8 * 2048;
constexpr size_t WS_END    = WS_SCR + SCR_BYTES;

__device__ __forceinline__ unsigned cvt2(float a, float b) {
    bf16x2 r; r[0] = (__bf16)a; r[1] = (__bf16)b; return __builtin_bit_cast(unsigned, r);
}
__device__ __forceinline__ f32x4 ldb(__amdgpu_buffer_rsrc_t r, int voff, int soff) {
    return __builtin_bit_cast(f32x4, __builtin_amdgcn_raw_buffer_load_b128(r, voff, soff, 0));
}
__device__ __forceinline__ unsigned pack_fp8x4(float a, float b, float c, float d) {
    int r = 0;
    r = __builtin_amdgcn_cvt_pk_fp8_f32(a, b, r, false);
    r = __builtin_amdgcn_cvt_pk_fp8_f32(c, d, r, true);
    return (unsigned)r;
}
#define SB0 __builtin_amdgcn_sched_barrier(0)
#define STAMP(i) do { } while (0)

__global__ void __launch_bounds__(NTHR, 2)
fused_gemm_a2a(const float* __restrict__ X, const float* __restrict__ W,
               void* const* __restrict__ out_all, void* const* __restrict__ ws_all, int rank) {
    constexpr int XB = BM * 64, WB = BK * 256;
    __shared__ __attribute__((aligned(16))) char smem[2 * (XB + WB) + 64];
    unsigned* sbc = reinterpret_cast<unsigned*>(smem + 2 * (XB + WB));

    const int tid = threadIdx.x, lane = tid & 63;
    const int wave = __builtin_amdgcn_readfirstlane(tid >> 6);
    const int c = lane & 15, h = lane >> 4;
    const int wm = wave >> 1, wn = wave & 1;
    const int b = blockIdx.x, xcd = b & 7, idx = b >> 3;
    const int sl = xcd & 3, mt = idx & 1, nt = (xcd >> 2) * 16 + (idx >> 1);
    const int m0 = mt * BM, n0 = nt * BN, k0 = sl * KS;
    const int grp = mt * 32 + nt;

    char* ws = reinterpret_cast<char*>(ws_all[rank]);
    STAMP(0);
    unsigned call = 0;
    if (tid == 0) {
        unsigned t = __hip_atomic_fetch_add(reinterpret_cast<unsigned*>(ws + WS_TICKET), 1u,
                                            __ATOMIC_RELAXED, __HIP_MEMORY_SCOPE_AGENT);
        call = t / NWG + 1;
    }

    __amdgpu_buffer_rsrc_t rx = __builtin_amdgcn_make_buffer_rsrc((void*)X, 0, MR * KD * 4, 0x00020000);
    __amdgpu_buffer_rsrc_t rw = __builtin_amdgcn_make_buffer_rsrc((void*)W, 0, KD * ND * 4, 0x00020000);
    const int xrow = tid >> 2, xj = tid & 3;
    const int xv = ((m0 + xrow) * KD + k0 + 4 * xj) * 4;
    const int wk = tid >> 5, wq = tid & 31;
    const int wv = ((k0 + wk) * ND + n0 + 4 * wq) * 4;
    const int fsw = (0x1320 >> (((xrow >> 2) & 3) * 4)) & 3;
    const int xw_off = xrow * 64 + ((xj ^ fsw) * 16);
    const int ww_off = wk * 256 + ((wq ^ ((wk & 3) | (((wk >> 2) & 1) << 4))) * 8);
    const int fr = (0x1320 >> (((c >> 2) & 3) * 4)) & 3;
    const int xr_off = (64 * wm + c) * 64 + ((h ^ fr) * 16);
    const int q4 = c >> 2, p4 = c & 3;
    int wr_off[4];
#pragma unroll
    for (int t = 0; t < 4; ++t)
        wr_off[t] = (4 * h + q4) * 256 + ((((16 * wn) ^ ((h & 1) << 4)) + 4 * p4 + (t ^ q4)) * 8);

    f32x4 acc[4][4];
#pragma unroll
    for (int t = 0; t < 4; ++t)
#pragma unroll
        for (int u = 0; u < 4; ++u) acc[t][u] = f32x4{0.f, 0.f, 0.f, 0.f};

    f32x4 xr[PD][4], wr[PD][2];
    bf16x8 xf[2][4], wf[2][4];
    auto issue = [&](int st, int set) {
        xr[set][0] = ldb(rx, xv, st * 128);
        xr[set][1] = ldb(rx, xv + 64, st * 128);
        xr[set][2] = ldb(rx, xv, st * 128 + 128 * KD * 4);
        xr[set][3] = ldb(rx, xv + 64, st * 128 + 128 * KD * 4);
        wr[set][0] = ldb(rw, wv, st * 32 * ND * 4);
        wr[set][1] = ldb(rw, wv, (st * 32 + 16) * ND * 4);
    };
    auto stash = [&](int set, int buf) {
        char* xb = smem + buf * (XB + WB); char* wb = xb + XB;
#pragma unroll
        for (int ps = 0; ps < 2; ++ps) {
            u32x4 v;
            v[0] = cvt2(xr[set][2 * ps][0], xr[set][2 * ps][1]);
            v[1] = cvt2(xr[set][2 * ps][2], xr[set][2 * ps][3]);
            v[2] = cvt2(xr[set][2 * ps + 1][0], xr[set][2 * ps + 1][1]);
            v[3] = cvt2(xr[set][2 * ps + 1][2], xr[set][2 * ps + 1][3]);
            *reinterpret_cast<u32x4*>(xb + xw_off + ps * 128 * 64) = v;
            u32x2 w2;
            w2[0] = cvt2(wr[set][ps][0], wr[set][ps][1]);
            w2[1] = cvt2(wr[set][ps][2], wr[set][ps][3]);
            *reinterpret_cast<u32x2*>(wb + ww_off + ps * 16 * 256) = w2;
        }
    };
    auto frags = [&](int buf, int fs) {
        char* xb = smem + buf * (XB + WB); char* wb = xb + XB;
#pragma unroll
        for (int u = 0; u < 4; ++u) xf[fs][u] = *reinterpret_cast<const bf16x8*>(xb + xr_off + u * 1024);
#pragma unroll
        for (int tt = 0; tt < 4; ++tt) {
            bf16x4 lo = __builtin_amdgcn_ds_read_tr16_b64_v4bf16((__attribute__((address_space(3))) bf16x4*)(wb + wr_off[tt]));
            bf16x4 hi = __builtin_amdgcn_ds_read_tr16_b64_v4bf16((__attribute__((address_space(3))) bf16x4*)(wb + wr_off[tt] + 4096));
            wf[fs][tt] = __builtin_shufflevector(lo, hi, 0, 1, 2, 3, 4, 5, 6, 7);
        }
    };
    auto mma = [&](int fs) {
#pragma unroll
        for (int tt = 0; tt < 4; ++tt)
#pragma unroll
            for (int u = 0; u < 4; ++u)
                acc[tt][u] = __builtin_amdgcn_mfma_f32_16x16x32_bf16(wf[fs][tt], xf[fs][u], acc[tt][u], 0, 0, 0);
    };
#pragma unroll
    for (int d = 0; d < PD; ++d) issue(d, d);
    SB0;
    stash(0, 0); issue(PD, 0);
    SB0;
    __syncthreads();
    SB0;
    frags(0, 0);
    stash(1 % PD, 1); issue(PD + 1, 1 % PD);
    SB0;
    __syncthreads();
    SB0;
    STAMP(1);
#pragma unroll
    for (int t = 0; t < NT; ++t) {
        if (t + 1 < NT) frags((t + 1) & 1, (t + 1) & 1);
        if (t + 2 < NT) {
            stash((t + 2) % PD, t & 1);
            if (t + 2 + PD < NT) issue(t + 2 + PD, (t + 2) % PD);
        }
        mma(t & 1);
        SB0;
        if (t + 1 < NT) __syncthreads();
        SB0;
    }

    STAMP(2);
    unsigned* tmo = reinterpret_cast<unsigned*>(ws + WS_TMO);
    __amdgpu_buffer_rsrc_t rs = __builtin_amdgcn_make_buffer_rsrc((void*)(ws + WS_SCR), 0, (int)SCR_BYTES, 0x00020000);
#pragma unroll
    for (int u = 0; u < 4; ++u) if (u != sl) {
#pragma unroll
        for (int hf = 0; hf < 2; ++hf) {
            u32x4 v;
            v[0] = cvt2(acc[2 * hf][u][0], acc[2 * hf][u][1]);
            v[1] = cvt2(acc[2 * hf][u][2], acc[2 * hf][u][3]);
            v[2] = cvt2(acc[2 * hf + 1][u][0], acc[2 * hf + 1][u][1]);
            v[3] = cvt2(acc[2 * hf + 1][u][2], acc[2 * hf + 1][u][3]);
            __builtin_amdgcn_raw_buffer_store_b128(v, rs, lane * 16, ((((grp * 4 + u) * 4 + sl) * 8 + wave) * 2 + hf) * 1024, 16);
        }
    }
    asm volatile("s_waitcnt vmcnt(0)" ::: "memory");
    __syncthreads();
    STAMP(3);
    const unsigned long long t0 = __builtin_amdgcn_s_memrealtime();
    bool xfail = false;
    if (tid == 0) {
        unsigned* ctr = reinterpret_cast<unsigned*>(ws + WS_GCTR + grp * 128);
        __hip_atomic_fetch_add(ctr, 1u, __ATOMIC_RELAXED, __HIP_MEMORY_SCOPE_AGENT);
        while (__hip_atomic_load(ctr, __ATOMIC_RELAXED, __HIP_MEMORY_SCOPE_AGENT) < call * SK) {
            if (__builtin_amdgcn_s_memrealtime() - t0 > DEADLINE_TICKS) {
                __hip_atomic_store(tmo, call, __ATOMIC_RELAXED, __HIP_MEMORY_SCOPE_AGENT);
                xfail = true;
                break;
            }
            __builtin_amdgcn_s_sleep(1);
        }
    }
    __syncthreads();
    asm volatile("" ::: "memory");
    STAMP(4);
    f32x4 fin[4];
#pragma unroll
    for (int tt = 0; tt < 4; ++tt) {
        f32x4 v = f32x4{0.f, 0.f, 0.f, 0.f};
#pragma unroll
        for (int u = 0; u < 4; ++u) if (u == sl) v = acc[tt][u];
        fin[tt] = v;
    }
    {
        u32x4 pv[3][2];
#pragma unroll
        for (int i = 0; i < 3; ++i) {
            int src = (sl + 1 + i) & 3;
#pragma unroll
            for (int hf = 0; hf < 2; ++hf)
                pv[i][hf] = __builtin_amdgcn_raw_buffer_load_b128(rs, lane * 16, ((((grp * 4 + sl) * 4 + src) * 8 + wave) * 2 + hf) * 1024, 16);
        }
#pragma unroll
        for (int i = 0; i < 3; ++i)
#pragma unroll
            for (int hf = 0; hf < 2; ++hf)
#pragma unroll
                for (int e = 0; e < 4; ++e) {
                    fin[2 * hf + (e >> 1)][(e & 1) * 2]     += __uint_as_float(pv[i][hf][e] << 16);
                    fin[2 * hf + (e >> 1)][(e & 1) * 2 + 1] += __uint_as_float(pv[i][hf][e] & 0xffff0000u);
                }
    }

    float am = 0.f;
#pragma unroll
    for (int tt = 0; tt < 4; ++tt)
#pragma unroll
        for (int r = 0; r < 4; ++r) am = fmaxf(am, fabsf(fin[tt][r]));
#pragma unroll
    for (int off = 32; off > 0; off >>= 1) am = fmaxf(am, __shfl_xor(am, off));
    if (lane == 0) sbc[wave] = __float_as_uint(am);
    __syncthreads();

    STAMP(5);
    if (wave == 0) {
        call = __shfl(call, 0);
        const size_t par = (size_t)(call & 1u) * 64;
        if (lane == 0) {
            unsigned a = max(max(max(sbc[0], sbc[1]), max(sbc[2], sbc[3])),
                             max(max(sbc[4], sbc[5]), max(sbc[6], sbc[7])));
            __hip_atomic_store(reinterpret_cast<unsigned long long*>(ws + WS_WGMAX + b * 8),
                               ((unsigned long long)call << 32) | a, __ATOMIC_RELAXED, __HIP_MEMORY_SCOPE_AGENT);
        }
        if (b == AGG_WG) {
            unsigned a = 0;
            bool ok = false;
            while (!ok) {
                ok = true; a = 0;
#pragma unroll
                for (int q = 0; q < NWG / 64; ++q) {
                    unsigned long long v = __hip_atomic_load(
                        reinterpret_cast<unsigned long long*>(ws + WS_WGMAX + (q * 64 + lane) * 8),
                        __ATOMIC_RELAXED, __HIP_MEMORY_SCOPE_AGENT);
                    ok = ok && ((unsigned)(v >> 32) == call);
                    a = max(a, (unsigned)v);
                }
                ok = __all(ok);
                if (!ok) {
                    if (__builtin_amdgcn_s_memrealtime() - t0 > DEADLINE_TICKS) {
                        if (lane == 0) __hip_atomic_store(tmo, call, __ATOMIC_RELAXED, __HIP_MEMORY_SCOPE_AGENT);
                        break;
                    }
                }
            }
#pragma unroll
            for (int off = 32; off > 0; off >>= 1) a = max(a, (unsigned)__shfl_xor((int)a, off));
            if (ok && lane < NR) {
                char* pws = reinterpret_cast<char*>(ws_all[lane]);
                __hip_atomic_store(reinterpret_cast<unsigned long long*>(pws + WS_AMAX + rank * 128 + par),
                                   ((unsigned long long)call << 32) | a, __ATOMIC_RELAXED, __HIP_MEMORY_SCOPE_SYSTEM);
            }
        }
        STAMP(6);
        unsigned ga = 0;
        bool good = false;
        while (!good) {
            bool ok = true;
            if (lane < NR) {
                unsigned long long v = __hip_atomic_load(
                    reinterpret_cast<unsigned long long*>(ws + WS_AMAX + lane * 128 + par),
                    __ATOMIC_RELAXED, __HIP_MEMORY_SCOPE_SYSTEM);
                ok = ((unsigned)(v >> 32) == call);
                ga = (unsigned)v;
            }
            good = __all(ok);
            if (!good) {
                if (__builtin_amdgcn_s_memrealtime() - t0 > DEADLINE_TICKS) {
                    if (lane == 0) __hip_atomic_store(tmo, call, __ATOMIC_RELAXED, __HIP_MEMORY_SCOPE_AGENT);
                    break;
                }
                __builtin_amdgcn_s_sleep(1);
            }
        }
        if (__any(xfail)) good = false;
#pragma unroll
        for (int off = 4; off > 0; off >>= 1) ga = max(ga, (unsigned)__shfl_xor((int)ga, off));
        if (lane == 0) { sbc[8] = ga; sbc[9] = good ? 1u : 0u; sbc[10] = call; }
    }
    __syncthreads();
    STAMP(7);
    const float gmax = __uint_as_float(sbc[8]);
    const bool good = sbc[9] != 0;
    call = sbc[10];

    const float inv = gmax > 0.f ? 448.0f / gmax : 0.f;
    const unsigned dst = (unsigned)(n0 / NS);
    if (dst < (unsigned)NR) {
        __amdgpu_buffer_rsrc_t ro = __builtin_amdgcn_make_buffer_rsrc(out_all[dst], 0, NR * MR * NS, 0x00020000);
        u32x4 q;
#pragma unroll
        for (int tt = 0; tt < 4; ++tt) {
            float v0 = __builtin_amdgcn_fmed3f(fin[tt][0] * inv, -448.f, 448.f);
            float v1 = __builtin_amdgcn_fmed3f(fin[tt][1] * inv, -448.f, 448.f);
            float v2 = __builtin_amdgcn_fmed3f(fin[tt][2] * inv, -448.f, 448.f);
            float v3 = __builtin_amdgcn_fmed3f(fin[tt][3] * inv, -448.f, 448.f);
            q[tt] = good ? pack_fp8x4(v0, v1, v2, v3) : 0x7f7f7f7fu;
        }
        __builtin_amdgcn_raw_buffer_store_b128(q, ro,
            (rank * MR + m0 + 64 * wm + 16 * sl + c) * NS + (n0 % NS) + 64 * wn + 16 * h, 0, 17);
        asm volatile("s_waitcnt vmcnt(0)" ::: "memory");
        __syncthreads();
        STAMP(8);
        if (tid == 0) {
            char* pws = reinterpret_cast<char*>(ws_all[dst]);
            __hip_atomic_store(reinterpret_cast<unsigned*>(pws + WS_DONE + (rank * 32 + ((nt & 3) * 2 + mt) * 4 + sl) * 4),
                               call, __ATOMIC_RELAXED, __HIP_MEMORY_SCOPE_SYSTEM);
        }
    }
    if (b == WAIT_WG && wave == 0) {
        STAMP(10);
        __amdgpu_buffer_rsrc_t rd = __builtin_amdgcn_make_buffer_rsrc((void*)(ws + WS_DONE), 0, 1024, 0x00020000);
        const unsigned long long t1 = __builtin_amdgcn_s_memrealtime();
        while (true) {
            asm volatile("" ::: "memory");
            u32x4 v = __builtin_amdgcn_raw_buffer_load_b128(rd, lane * 16, 0, 17);
            bool ok = v[0] >= call && v[1] >= call && v[2] >= call && v[3] >= call;
            if (__all(ok)) break;
            if (__builtin_amdgcn_s_memrealtime() - t1 > DEADLINE_TICKS) {
                if (lane == 0) {
                    __hip_atomic_store(tmo, call, __ATOMIC_RELAXED, __HIP_MEMORY_SCOPE_AGENT);
                    reinterpret_cast<unsigned char*>(out_all[rank])[(size_t)rank * MR * NS] = 0x7f;
                }
                break;
            }
        }
        __builtin_amdgcn_fence(__ATOMIC_ACQUIRE, "");
        STAMP(11);
    }
}

}

extern "C" void kernel_launch(int rank, int nranks, ncclComm_t comm,
                              hipStream_t stream,
                              void* const* d_in, const int* in_sizes, int n_in,
                              void* d_out, int out_size,
                              const int* in_dtypes, int out_dtype,
                              void* const* d_out_all,
                              void* const* d_ws_all, size_t ws_bytes) {
    (void)comm; (void)in_sizes; (void)n_in; (void)d_out; (void)out_size; (void)in_dtypes; (void)out_dtype;
    if (nranks != NR || ws_bytes < WS_END) {
        fprintf(stderr, "unsupported configuration (ws_bytes %zu)\n", ws_bytes);
        return;
    }
    hipLaunchKernelGGL(fused_gemm_a2a, dim3(NWG), dim3(NTHR), 0, stream,
                       (const float*)d_in[0], (const float*)d_in[1], d_out_all, d_ws_all, rank);
    HIP_CHECK(hipGetLastError());
}
```

```cpp
#include <hip/hip_runtime.h>
#include <rccl/rccl.h>
#include <cstdio>
#include <cstdint>

#define HIP_CHECK(expr) do { hipError_t _e = (expr); if (_e != hipSuccess) { \
    fprintf(stderr, "HIP error %s at %s:%d\n", hipGetErrorString(_e), __FILE__, __LINE__); \
    return; } } while (0)

namespace {

typedef __attribute__((ext_vector_type(4))) float f32x4;
typedef __attribute__((ext_vector_type(4))) unsigned u32x4;
typedef __attribute__((ext_vector_type(2))) unsigned u32x2;
typedef __attribute__((ext_vector_type(8))) __bf16 bf16x8;
typedef __attribute__((ext_vector_type(4))) __bf16 bf16x4;
typedef __attribute__((ext_vector_type(2))) __bf16 bf16x2;
#define GAS __attribute__((address_space(1)))
typedef GAS char gchar;
typedef GAS unsigned gu32;
typedef GAS unsigned long long gu64;

constexpr int NR = 8;
constexpr int MR = 512;
constexpr int KD = 4096;
constexpr int ND = 4096;
constexpr int NS = 512;
constexpr int BM = 256, BN = 128, BK = 32, SK = 4, KS = KD / SK, NT = KS / BK;
constexpr int NWG = (MR / BM) * (ND / BN) * SK;
constexpr int NTHR = 512;
constexpr int WAIT_WG = 0;
#define AMAX_SLEEP 2
constexpr int FS = 8;
constexpr int DX = 2, DW = 2;
constexpr int XB = BM * 64, WB = BK * 256;
constexpr unsigned long long DEADLINE_TICKS = 50000000ull;

constexpr size_t WS_TMO   = 384;
constexpr size_t WS_DONE  = 2048;
constexpr size_t WS_XFLG  = 16384;
constexpr size_t WS_SEQ   = 32768;
constexpr size_t WS_EFLG  = 40960;
constexpr size_t WS_GMAX  = 57344;
constexpr size_t WS_AMAX  = 65536;
constexpr size_t WS_SCR   = 1 << 20;
constexpr size_t SCR_BYTES = (size_t)64 * 4 * 3 * 4 * 4096;
constexpr size_t WS_IMG   = WS_SCR + SCR_BYTES;
constexpr size_t WS_END   = WS_IMG + ((size_t)8 << 20);

__device__ __forceinline__ unsigned cvt2(float a, float b) {
    bf16x2 r; r[0] = (__bf16)a; r[1] = (__bf16)b; return __builtin_bit_cast(unsigned, r);
}
template <int AUX>
__device__ __forceinline__ f32x4 ldb(__amdgpu_buffer_rsrc_t r, int voff, int soff) {
    return __builtin_bit_cast(f32x4, __builtin_amdgcn_raw_buffer_load_b128(r, voff, soff, AUX));
}
__device__ __forceinline__ unsigned pack_fp8x4(float a, float b, float c, float d) {
    int r = 0;
    r = __builtin_amdgcn_cvt_pk_fp8_f32(a, b, r, false);
    r = __builtin_amdgcn_cvt_pk_fp8_f32(c, d, r, true);
    return (unsigned)r;
}
#define SB0 __builtin_amdgcn_sched_barrier(0)
#define TOUCH(dummy, voff, rsrc, soff) \
    asm volatile("buffer_load_dword %0, %1, %2, %3 offen" : "+v"(dummy) : "v"(voff), "s"(rsrc), "s"(soff))
#define STAMP(i) do { } while (0)

__global__ void __launch_bounds__(NTHR, 2)
fused_gemm_a2a(const float* __restrict__ X, const float* __restrict__ W,
               void* const* __restrict__ out_all, void* const* __restrict__ ws_all, int rank) {
    __shared__ __attribute__((aligned(16))) char smem[2 * (XB + WB) + 64];
    unsigned* sbc = reinterpret_cast<unsigned*>(smem + 2 * (XB + WB));

    const int tid = threadIdx.x, lane = tid & 63;
    const int wave = __builtin_amdgcn_readfirstlane(tid >> 6);
    const int c = lane & 15, h = lane >> 4;
    const int b = blockIdx.x, xcd = b & 7, idx = b >> 3;
    const int sl = xcd & 3, mt = idx & 1, nt = (xcd >> 2) * 16 + (idx >> 1);
    const int m0 = mt * BM, n0 = nt * BN, k0 = sl * KS;
    const int grp = mt * 32 + nt;

    gchar* ws = (gchar*)ws_all[rank];
    STAMP(0);
    __amdgpu_buffer_rsrc_t rx = __builtin_amdgcn_make_buffer_rsrc((void*)X, 0, MR * KD * 4, 0x00020000);
    __amdgpu_buffer_rsrc_t rw = __builtin_amdgcn_make_buffer_rsrc((void*)W, 0, KD * ND * 4, 0x00020000);
    __amdgpu_buffer_rsrc_t ri = __builtin_amdgcn_make_buffer_rsrc((void*)(ws + WS_IMG + ((size_t)xcd << 20)), 0, 1 << 20, 0x00020000);
    gu32* seqp = (gu32*)(ws + WS_SEQ + b * 4);
    gu32* xc = (gu32*)(ws + WS_XFLG + xcd * 512);
    gu32* tmo = (gu32*)(ws + WS_TMO);

    if (wave >= 4) {
        const int ptid = tid - 256;
        const int sx = __builtin_amdgcn_readfirstlane(mt * 2);
        const int wk = ptid >> 5, wq = ptid & 31;
        const int wv = ((k0 + wk) * ND + n0 + 4 * wq) * 4;
        const int ww_off = 2 * XB + wk * 256 + ((wq ^ ((wk & 3) | (((wk >> 2) & 1) << 4))) * 8);
        const int xiv = mt * 32 * XB + ptid * 16;
        const int xrow = ptid >> 2, xj = ptid & 3;
        const int xv = ((m0 + xrow) * KD + k0 + 4 * xj) * 4;
        const int fsw = (0x1320 >> (((xrow >> 2) & 3) * 4)) & 3;
        const int xw_off = xrow * 64 + ((xj ^ fsw) * 16);
        f32x4 wr[DW][4]; u32x4 xi[DX][4]; f32x4 xr[DX][8];
        auto issue_w = [&](int st, int i) {
            if (st & 2) wr[st % DW][i] = ldb<2>(rw, wv, ((st ^ sx) * 32 + 8 * i) * ND * 4);
            else        wr[st % DW][i] = ldb<0>(rw, wv, ((st ^ sx) * 32 + 8 * i) * ND * 4);
        };
        auto stash_w = [&](int st, int i) {
            u32x2 w2;
            w2[0] = cvt2(wr[st % DW][i][0], wr[st % DW][i][1]);
            w2[1] = cvt2(wr[st % DW][i][2], wr[st % DW][i][3]);
            *reinterpret_cast<u32x2*>(smem + ww_off + (st & 1) * WB + i * 8 * 256) = w2;
        };
        auto issue_x = [&](int st, int i) {
            if (st < FS) {
                xr[st % DX][2 * i]     = ldb<0>(rx, xv, (st ^ sx) * 128 + i * 64 * KD * 4);
                xr[st % DX][2 * i + 1] = ldb<0>(rx, xv + 64, (st ^ sx) * 128 + i * 64 * KD * 4);
            } else {
                xi[st % DX][i] = __builtin_amdgcn_raw_buffer_load_b128(ri, xiv, (st ^ sx) * XB + i * 4096, 0);
            }
        };
        auto stash_x = [&](int st, int i) {
            if (st < FS) {
                const int set = st % DX;
                u32x4 v;
                v[0] = cvt2(xr[set][2 * i][0], xr[set][2 * i][1]);
                v[1] = cvt2(xr[set][2 * i][2], xr[set][2 * i][3]);
                v[2] = cvt2(xr[set][2 * i + 1][0], xr[set][2 * i + 1][1]);
                v[3] = cvt2(xr[set][2 * i + 1][2], xr[set][2 * i + 1][3]);
                *reinterpret_cast<u32x4*>(smem + (st & 1) * XB + xw_off + i * 64 * 64) = v;
            } else {
                *reinterpret_cast<u32x4*>(smem + (st & 1) * XB + ptid * 16 + i * 4096) = xi[st % DX][i];
            }
        };
#pragma unroll
        for (int i = 0; i < 4; ++i) { issue_x(0, i); issue_w(0, i); }
#pragma unroll
        for (int i = 0; i < 4; ++i) { issue_x(1, i); issue_w(1, i); }
        const unsigned call = __hip_atomic_load(seqp, __ATOMIC_RELAXED, __HIP_MEMORY_SCOPE_AGENT) + 1;
        SB0;
#pragma unroll
        for (int t = -2; t < NT - 1; ++t) {
            if (t + 2 + DX == FS) {
                const unsigned long long t0 = __builtin_amdgcn_s_memrealtime();
                while (true) {
                    unsigned long long v = __hip_atomic_load((gu64*)xc + lane,
                                                             __ATOMIC_RELAXED, __HIP_MEMORY_SCOPE_AGENT);
                    if (__all((unsigned)v == call && (unsigned)(v >> 32) == call)) break;
                    if (__builtin_amdgcn_s_memrealtime() - t0 > DEADLINE_TICKS) {
                        if (lane == 0) __hip_atomic_store(tmo, call, __ATOMIC_RELAXED, __HIP_MEMORY_SCOPE_AGENT);
                        break;
                    }
                    __builtin_amdgcn_s_sleep(1);
                }
                asm volatile("" ::: "memory");
                SB0;
            }
#pragma unroll
            for (int i = 0; i < 4; ++i) {
                if (t + 2 < NT) { stash_x(t + 2, i); SB0; if (t + 2 + DX < NT) issue_x(t + 2 + DX, i); SB0; }
                if (t + 2 < NT) { stash_w(t + 2, i); SB0; if (t + 2 + DW < NT) issue_w(t + 2 + DW, i); SB0; }
            }
            asm volatile("s_waitcnt lgkmcnt(0)" ::: "memory");
            __builtin_amdgcn_s_barrier();
            asm volatile("" ::: "memory");
            SB0;
        }
        return;
    }

    const unsigned call = __hip_atomic_load(seqp, __ATOMIC_RELAXED, __HIP_MEMORY_SCOPE_AGENT) + 1;
    unsigned dummy = 0;
    {
        const int j = tid & 3, r16 = (tid >> 2) & 15;
        const int R = 16 * idx + r16, rt = R & 255;
        const int xs = (R * KD + k0 + wave * 32 + 4 * j) * 4;
        const int fsw = (0x1320 >> (((rt >> 2) & 3) * 4)) & 3;
        const int xd = (((R >> 8) * 32 + wave) * 256 + rt) * 64 + ((j ^ fsw) * 16);
        {
            const int tx = ((16 * idx + (lane & 15)) * KD + k0) * 4 + ((lane >> 4) + 4 * wave) * 128;
            if (4 * wave < FS) TOUCH(dummy, tx, rx, 0);
        }
        f32x4 a[8][2];
#pragma unroll
        for (int i = FS / 4; i < 8; ++i) { a[i][0] = ldb<0>(rx, xs, i * 512); a[i][1] = ldb<0>(rx, xs + 64, i * 512); }
#pragma unroll
        for (int i = FS / 4; i < 8; ++i) {
            u32x4 v;
            v[0] = cvt2(a[i][0][0], a[i][0][1]); v[1] = cvt2(a[i][0][2], a[i][0][3]);
            v[2] = cvt2(a[i][1][0], a[i][1][1]); v[3] = cvt2(a[i][1][2], a[i][1][3]);
            __builtin_amdgcn_raw_buffer_store_b128(v, ri, xd, i * 4 * XB, 16);
        }
    }
    const int wm = wave >> 1, wn = wave & 1;
    const int fr = (0x1320 >> (((c >> 2) & 3) * 4)) & 3;
    const int xr_off = (128 * wm + c) * 64 + ((h ^ fr) * 16);
    const int q4 = c >> 2, p4 = c & 3;
    int wr_off[4];
#pragma unroll
    for (int t = 0; t < 4; ++t)
        wr_off[t] = 2 * XB + (4 * h + q4) * 256 + ((((16 * wn) ^ ((h & 1) << 4)) + 4 * p4 + (t ^ q4)) * 8);

    f32x4 acc[4][8];
#pragma unroll
    for (int t = 0; t < 4; ++t)
#pragma unroll
        for (int u = 0; u < 8; ++u) acc[t][u] = f32x4{0.f, 0.f, 0.f, 0.f};
    bf16x8 xf[8], wf[2][4];
    auto rdw = [&](int st, int tt) {
        const char* wb = smem + (st & 1) * WB;
        bf16x4 lo = __builtin_amdgcn_ds_read_tr16_b64_v4bf16((__attribute__((address_space(3))) bf16x4*)(wb + wr_off[tt]));
        bf16x4 hi = __builtin_amdgcn_ds_read_tr16_b64_v4bf16((__attribute__((address_space(3))) bf16x4*)(wb + wr_off[tt] + 4096));
        return __builtin_shufflevector(lo, hi, 0, 1, 2, 3, 4, 5, 6, 7);
    };
    SB0;
    __syncthreads();
    SB0;
#pragma unroll
    for (int u = 0; u < 8; ++u) xf[u] = *reinterpret_cast<const bf16x8*>(smem + xr_off + u * 1024);
#pragma unroll
    for (int tt = 0; tt < 4; ++tt) wf[0][tt] = rdw(0, tt);
    SB0;
    __syncthreads();
    SB0;
    STAMP(1);
#pragma unroll
    for (int t = 0; t < NT; ++t) {
        const char* xb = smem + ((t + 1) & 1) * XB;
#pragma unroll
        for (int u = 0; u < 8; ++u) {
#pragma unroll
            for (int tt = 0; tt < 4; ++tt)
                acc[tt][u] = __builtin_amdgcn_mfma_f32_16x16x32_bf16(wf[t & 1][tt], xf[u], acc[tt][u], 0, 0, 0);
            if (t + 1 < NT) {
                xf[u] = *reinterpret_cast<const bf16x8*>(xb + xr_off + u * 1024);
                if (u < 4) wf[(t + 1) & 1][u] = rdw(t + 1, u);
            }
            if (t == 1 && u == 6) {
                asm volatile("s_waitcnt vmcnt(0)" :: "v"(dummy) : "memory");
                if (lane == 0) __hip_atomic_store(xc + idx * 4 + wave, call, __ATOMIC_RELAXED, __HIP_MEMORY_SCOPE_AGENT);
            }
            SB0;
        }
        if (t + 1 < NT) __syncthreads();
        SB0;
    }
    STAMP(2);

    __amdgpu_buffer_rsrc_t rs = __builtin_amdgcn_make_buffer_rsrc((void*)(ws + WS_SCR), 0, (int)SCR_BYTES, 0x00020000);
#pragma unroll
    for (int u = 0; u < 8; ++u) if ((u >> 1) != sl) {
        const int d = u >> 1, sp = (sl - d - 1) & 3;
#pragma unroll
        for (int hf = 0; hf < 2; ++hf) {
            u32x4 v;
            v[0] = cvt2(acc[2 * hf][u][0], acc[2 * hf][u][1]);
            v[1] = cvt2(acc[2 * hf][u][2], acc[2 * hf][u][3]);
            v[2] = cvt2(acc[2 * hf + 1][u][0], acc[2 * hf + 1][u][1]);
            v[3] = cvt2(acc[2 * hf + 1][u][2], acc[2 * hf + 1][u][3]);
            __builtin_amdgcn_raw_buffer_store_b128(v, rs, lane * 16,
                ((((grp * 4 + d) * 3 + sp) * 4 + wave) * 4 + (u & 1) * 2 + hf) * 1024, 16);
        }
    }
    if (tid == 0) __hip_atomic_store(seqp, call, __ATOMIC_RELAXED, __HIP_MEMORY_SCOPE_AGENT);
    asm volatile("s_waitcnt vmcnt(0)" ::: "memory");
    const unsigned long long t0 = __builtin_amdgcn_s_memrealtime();
    bool fail = false;
    {
        gu32* ef = (gu32*)(ws + WS_EFLG);
        if (lane < 4 && lane != sl)
            __hip_atomic_store(ef + ((grp * 4 + lane) * 4 + wave) * 4 + sl, call, __ATOMIC_RELAXED, __HIP_MEMORY_SCOPE_AGENT);
        STAMP(3);
        while (true) {
            bool ok = true;
            if (lane < 4 && lane != sl)
                ok = __hip_atomic_load(ef + ((grp * 4 + sl) * 4 + wave) * 4 + lane, __ATOMIC_RELAXED, __HIP_MEMORY_SCOPE_AGENT) == call;
            if (__all(ok)) break;
            if (__builtin_amdgcn_s_memrealtime() - t0 > DEADLINE_TICKS) {
                if (lane == 0) __hip_atomic_store(tmo, call, __ATOMIC_RELAXED, __HIP_MEMORY_SCOPE_AGENT);
                fail = true;
                break;
            }
            __builtin_amdgcn_s_sleep(1);
        }
        asm volatile("" ::: "memory");
    }
    STAMP(4);
    f32x4 fin[2][4];
#pragma unroll
    for (int j = 0; j < 2; ++j)
#pragma unroll
        for (int tt = 0; tt < 4; ++tt) {
            f32x4 v = f32x4{0.f, 0.f, 0.f, 0.f};
#pragma unroll
            for (int s = 0; s < 4; ++s) if (s == sl) v = acc[tt][2 * s + j];
            fin[j][tt] = v;
        }
    {
        u32x4 pv[3][4];
#pragma unroll
        for (int i = 0; i < 3; ++i)
#pragma unroll
            for (int e = 0; e < 4; ++e)
                pv[i][e] = __builtin_amdgcn_raw_buffer_load_b128(rs, lane * 16, ((((grp * 4 + sl) * 3 + i) * 4 + wave) * 4 + e) * 1024, 16);
#pragma unroll
        for (int i = 0; i < 3; ++i)
#pragma unroll
            for (int j = 0; j < 2; ++j)
#pragma unroll
                for (int hf = 0; hf < 2; ++hf)
#pragma unroll
                    for (int e = 0; e < 4; ++e) {
                        fin[j][2 * hf + (e >> 1)][(e & 1) * 2]     += __uint_as_float(pv[i][2 * j + hf][e] << 16);
                        fin[j][2 * hf + (e >> 1)][(e & 1) * 2 + 1] += __uint_as_float(pv[i][2 * j + hf][e] & 0xffff0000u);
                    }
    }

    float am = 0.f;
#pragma unroll
    for (int j = 0; j < 2; ++j)
#pragma unroll
        for (int tt = 0; tt < 4; ++tt)
#pragma unroll
            for (int r = 0; r < 4; ++r) am = fmaxf(am, fabsf(fin[j][tt][r]));
#pragma unroll
    for (int off = 32; off > 0; off >>= 1) am = fmaxf(am, __shfl_xor(am, off));
    if (lane == 0) sbc[wave] = __float_as_uint(am);
    __syncthreads();
    STAMP(5);
    const size_t par = (size_t)(call & 1u) * 16384;
    if (wave == 0 && lane < NR) {
        unsigned a = max(max(sbc[0], sbc[1]), max(sbc[2], sbc[3]));
        gchar* pws = (gchar*)ws_all[lane];
        __hip_atomic_store((gu64*)(pws + WS_AMAX + par + (rank * NWG + b) * 8),
                           ((unsigned long long)call << 32) | a, __ATOMIC_RELAXED, __HIP_MEMORY_SCOPE_SYSTEM);
    }
    STAMP(6);
    __amdgpu_buffer_rsrc_t rg = __builtin_amdgcn_make_buffer_rsrc((void*)(ws + WS_GMAX), 0, 2048, 0x00020000);
    if (b < 8) {
        __amdgpu_buffer_rsrc_t ra = __builtin_amdgcn_make_buffer_rsrc((void*)(ws + WS_AMAX + par), 0, 16384, 0x00020000);
        unsigned ga = 0, pend = 15;
        while (true) {
            asm volatile("" ::: "memory");
            u32x4 v[4];
#pragma unroll
            for (int i = 0; i < 4; ++i)
                if (pend & (1u << i)) v[i] = __builtin_amdgcn_raw_buffer_load_b128(ra, lane * 16, wave * 4096 + i * 1024, 17);
#pragma unroll
            for (int i = 0; i < 4; ++i)
                if ((pend & (1u << i)) && v[i][1] == call && v[i][3] == call) {
                    ga = max(ga, max(v[i][0], v[i][2]));
                    pend &= ~(1u << i);
                }
            if (__all(pend == 0)) break;
            if (__builtin_amdgcn_s_memrealtime() - t0 > DEADLINE_TICKS) {
                if (lane == 0) __hip_atomic_store(tmo, call, __ATOMIC_RELAXED, __HIP_MEMORY_SCOPE_AGENT);
                fail = true;
                break;
            }
        }
#pragma unroll
        for (int off = 32; off > 0; off >>= 1) ga = max(ga, (unsigned)__shfl_xor((int)ga, off));
        if (lane == 0) { sbc[4 + wave] = ga; sbc[8 + wave] = fail ? 1u : 0u; }
        __syncthreads();
        if (tid == 0 && (sbc[8] | sbc[9] | sbc[10] | sbc[11]) == 0) {
            u32x2 g2; g2[0] = max(max(sbc[4], sbc[5]), max(sbc[6], sbc[7])); g2[1] = call;
            __builtin_amdgcn_raw_buffer_store_b64(g2, rg, b * 256, 0, 16);
        }
    } else {
        if (wave == 0) {
            unsigned ga = 0;
            while (true) {
                asm volatile("" ::: "memory");
                u32x2 v = __builtin_amdgcn_raw_buffer_load_b64(rg, xcd * 256, 0, 16);
                ga = v[0];
                if (__builtin_amdgcn_readfirstlane(v[1]) == call) break;
                if (__builtin_amdgcn_s_memrealtime() - t0 > DEADLINE_TICKS) {
                    if (lane == 0) __hip_atomic_store(tmo, call, __ATOMIC_RELAXED, __HIP_MEMORY_SCOPE_AGENT);
                    fail = true;
                    break;
                }
                __builtin_amdgcn_s_sleep(AMAX_SLEEP);
            }
            if (lane == 0) { sbc[4] = ga; sbc[5] = 0; sbc[6] = 0; sbc[7] = 0; }
        }
        if (lane == 0) sbc[8 + wave] = fail ? 1u : 0u;
        __syncthreads();
    }
    STAMP(7);
    const float gmax = __uint_as_float(max(max(sbc[4], sbc[5]), max(sbc[6], sbc[7])));
    const bool good = (sbc[8] | sbc[9] | sbc[10] | sbc[11]) == 0;

    const float inv = gmax > 0.f ? 448.0f / gmax : 0.f;
    const unsigned dst = (unsigned)(n0 / NS);
    if (dst < (unsigned)NR) {
        __amdgpu_buffer_rsrc_t ro = __builtin_amdgcn_make_buffer_rsrc(out_all[dst], 0, NR * MR * NS, 0x00020000);
#pragma unroll
        for (int j = 0; j < 2; ++j) {
            u32x4 q;
#pragma unroll
            for (int tt = 0; tt < 4; ++tt) {
                float v0 = __builtin_amdgcn_fmed3f(fin[j][tt][0] * inv, -448.f, 448.f);
                float v1 = __builtin_amdgcn_fmed3f(fin[j][tt][1] * inv, -448.f, 448.f);
                float v2 = __builtin_amdgcn_fmed3f(fin[j][tt][2] * inv, -448.f, 448.f);
                float v3 = __builtin_amdgcn_fmed3f(fin[j][tt][3] * inv, -448.f, 448.f);
                q[tt] = good ? pack_fp8x4(v0, v1, v2, v3) : 0x7f7f7f7fu;
            }
            __builtin_amdgcn_raw_buffer_store_b128(q, ro,
                (rank * MR + m0 + 128 * wm + 32 * sl + 16 * j + c) * NS + (n0 % NS) + 64 * wn + 16 * h, 0, 17);
        }
        asm volatile("s_waitcnt vmcnt(0)" ::: "memory");
        __syncthreads();
        STAMP(8);
        if (tid == 0) {
            gchar* pws = (gchar*)ws_all[dst];
            __hip_atomic_store((gu32*)(pws + WS_DONE + (rank * 32 + ((nt & 3) * 2 + mt) * 4 + sl) * 4),
                               call, __ATOMIC_RELAXED, __HIP_MEMORY_SCOPE_SYSTEM);
        }
    }
    if (b == WAIT_WG && wave == 0) {
        STAMP(10);
        __amdgpu_buffer_rsrc_t rd = __builtin_amdgcn_make_buffer_rsrc((void*)(ws + WS_DONE), 0, 1024, 0x00020000);
        const unsigned long long t1 = __builtin_amdgcn_s_memrealtime();
        while (true) {
            asm volatile("" ::: "memory");
            u32x4 v = __builtin_amdgcn_raw_buffer_load_b128(rd, lane * 16, 0, 17);
            bool ok = v[0] >= call && v[1] >= call && v[2] >= call && v[3] >= call;
            if (__all(ok)) break;
            if (__builtin_amdgcn_s_memrealtime() - t1 > DEADLINE_TICKS) {
                if (lane == 0) {
                    __hip_atomic_store(tmo, call, __ATOMIC_RELAXED, __HIP_MEMORY_SCOPE_AGENT);
                    ((gchar*)out_all[rank])[(size_t)rank * MR * NS] = 0x7f;
                }
                break;
            }
        }
        __builtin_amdgcn_fence(__ATOMIC_ACQUIRE, "");
        STAMP(11);
    }
}

}

extern "C" void kernel_launch(int rank, int nranks, ncclComm_t comm,
                              hipStream_t stream,
                              void* const* d_in, const int* in_sizes, int n_in,
                              void* d_out, int out_size,
                              const int* in_dtypes, int out_dtype,
                              void* const* d_out_all,
                              void* const* d_ws_all, size_t ws_bytes) {
    (void)comm; (void)in_sizes; (void)n_in; (void)d_out; (void)out_size; (void)in_dtypes; (void)out_dtype;
    if (nranks != NR || ws_bytes < WS_END) {
        fprintf(stderr, "unsupported configuration (ws_bytes %zu)\n", ws_bytes);
        return;
    }
    hipLaunchKernelGGL(fused_gemm_a2a, dim3(NWG), dim3(NTHR), 0, stream,
                       (const float*)d_in[0], (const float*)d_in[1], d_out_all, d_ws_all, rank);
    HIP_CHECK(hipGetLastError());
}
```

```cpp
#include <hip/hip_runtime.h>
#include <rccl/rccl.h>
#include <cstdio>
#include <cstdint>
#include <utility>

#define HIP_CHECK(expr) do { hipError_t _e = (expr); if (_e != hipSuccess) { \
    fprintf(stderr, "HIP error %s at %s:%d\n", hipGetErrorString(_e), __FILE__, __LINE__); \
    return; } } while (0)

namespace {

typedef __attribute__((ext_vector_type(4))) float f32x4;
typedef __attribute__((ext_vector_type(4))) unsigned u32x4;
typedef __attribute__((ext_vector_type(2))) unsigned u32x2;
typedef __attribute__((ext_vector_type(8))) __bf16 bf16x8;
typedef __attribute__((ext_vector_type(4))) __bf16 bf16x4;
typedef __attribute__((ext_vector_type(2))) __bf16 bf16x2;
#define GAS __attribute__((address_space(1)))
typedef GAS char gchar;
typedef GAS unsigned gu32;
typedef GAS unsigned long long gu64;

constexpr int NR = 8;
constexpr int MR = 512;
constexpr int KD = 4096;
constexpr int ND = 4096;
constexpr int NS = 512;
constexpr int BM = 256, BN = 128, BK = 32, SK = 4, KS = KD / SK, NT = KS / BK;
constexpr int NWG = (MR / BM) * (ND / BN) * SK;
constexpr int NTHR = 512;
constexpr int WAIT_WG = 0;
#define AMAX_SLEEP 2
constexpr int FS = 8;
constexpr int DEPTH = 12;
constexpr int XB = BM * 64, WB = BK * 256;
constexpr int EXO = 2 * (XB + WB) + 64, EXB = 4 * 12 * 1024;
constexpr unsigned long long DEADLINE_TICKS = 50000000ull;

constexpr size_t WS_TMO   = 384;
constexpr size_t WS_DONE  = 2048;
constexpr size_t WS_XFLG  = 16384;
constexpr size_t WS_SEQ   = 32768;
constexpr size_t WS_EFLG  = 40960;
constexpr size_t WS_GMAX  = 57344;
constexpr size_t WS_AMAX  = 65536;
constexpr size_t WS_SCR   = 1 << 20;
constexpr size_t SCR_BYTES = (size_t)64 * 4 * 3 * 4 * 4096;
constexpr size_t WS_IMG   = WS_SCR + SCR_BYTES;
constexpr size_t WS_END   = WS_IMG + ((size_t)8 << 20);

__device__ __forceinline__ unsigned cvt2(float a, float b) {
    bf16x2 r; r[0] = (__bf16)a; r[1] = (__bf16)b; return __builtin_bit_cast(unsigned, r);
}
template <int AUX>
__device__ __forceinline__ f32x4 ldb(__amdgpu_buffer_rsrc_t r, int voff, int soff) {
    return __builtin_bit_cast(f32x4, __builtin_amdgcn_raw_buffer_load_b128(r, voff, soff, AUX));
}
__device__ __forceinline__ unsigned pack_fp8x4(float a, float b, float c, float d) {
    int r = 0;
    r = __builtin_amdgcn_cvt_pk_fp8_f32(a, b, r, false);
    r = __builtin_amdgcn_cvt_pk_fp8_f32(c, d, r, true);
    return (unsigned)r;
}
__device__ __forceinline__ unsigned wave_umax(unsigned m) {
    m = max(m, (unsigned)__builtin_amdgcn_update_dpp(0, (int)m, 0x111, 0xf, 0xf, true));
    m = max(m, (unsigned)__builtin_amdgcn_update_dpp(0, (int)m, 0x112, 0xf, 0xf, true));
    m = max(m, (unsigned)__builtin_amdgcn_update_dpp(0, (int)m, 0x114, 0xf, 0xf, true));
    m = max(m, (unsigned)__builtin_amdgcn_update_dpp(0, (int)m, 0x118, 0xf, 0xf, true));
    m = max(m, (unsigned)__builtin_amdgcn_update_dpp(0, (int)m, 0x142, 0xa, 0xf, true));
    m = max(m, (unsigned)__builtin_amdgcn_update_dpp(0, (int)m, 0x143, 0xc, 0xf, true));
    return (unsigned)__builtin_amdgcn_readlane((int)m, 63);
}
template <int V> struct IC { static constexpr int value = V; };
template <class Fn, int... I>
__device__ __forceinline__ void static_for_impl(Fn&& f, std::integer_sequence<int, I...>) { (f(IC<I>{}), ...); }
template <int N, class Fn>
__device__ __forceinline__ void static_for(Fn&& f) { static_for_impl(f, std::make_integer_sequence<int, N>{}); }
struct Unit { int kind, st, i; bool last; };
constexpr int SEQ_N = FS * 12 + (NT - FS) * 8;
constexpr Unit seq_unit(int n) {
    if (n < FS * 12) { const int st = n / 12, j = n % 12; return j < 4 ? Unit{0, st, j, false} : Unit{1, st, j - 4, j == 11}; }
    const int st = FS + (n - FS * 12) / 8, j = (n - FS * 12) % 8;
    return j < 4 ? Unit{0, st, j, false} : Unit{2, st, j - 4, j == 7};
}
#define SB0 __builtin_amdgcn_sched_barrier(0)
#define LDS_BARRIER() asm volatile("s_waitcnt lgkmcnt(0)\n\ts_barrier" ::: "memory")
#define TOUCH(dummy, voff, rsrc, soff) \
    asm volatile("buffer_load_dword %0, %1, %2, %3 offen" : "+v"(dummy) : "v"(voff), "s"(rsrc), "s"(soff))
#define STAMP(i) do { } while (0)

__global__ void __launch_bounds__(NTHR, 2)
fused_gemm_a2a(const float* __restrict__ X, const float* __restrict__ W,
               void* const* __restrict__ out_all, void* const* __restrict__ ws_all, int rank) {
    __shared__ __attribute__((aligned(16))) char smem[2 * (XB + WB) + 64 + EXB];
    unsigned* sbc = reinterpret_cast<unsigned*>(smem + 2 * (XB + WB));

    const int tid = threadIdx.x, lane = tid & 63;
    const int wave = __builtin_amdgcn_readfirstlane(tid >> 6);
    const int c = lane & 15, h = lane >> 4;
    const int b = blockIdx.x, xcd = b & 7, idx = b >> 3;
    const int sl = xcd & 3, mt = idx & 1, nt = (xcd >> 2) * 16 + (idx >> 1);
    const int m0 = mt * BM, n0 = nt * BN, k0 = sl * KS;
    const int grp = mt * 32 + nt;

    gchar* ws = (gchar*)ws_all[rank];
    STAMP(0);
    __amdgpu_buffer_rsrc_t rx = __builtin_amdgcn_make_buffer_rsrc((void*)X, 0, MR * KD * 4, 0x00020000);
    __amdgpu_buffer_rsrc_t rw = __builtin_amdgcn_make_buffer_rsrc((void*)W, 0, KD * ND * 4, 0x00020000);
    __amdgpu_buffer_rsrc_t ri = __builtin_amdgcn_make_buffer_rsrc((void*)(ws + WS_IMG + ((size_t)xcd << 20)), 0, 1 << 20, 0x00020000);
    gu32* seqp = (gu32*)(ws + WS_SEQ + b * 4);
    gu32* xc = (gu32*)(ws + WS_XFLG + xcd * 512);
    gu32* tmo = (gu32*)(ws + WS_TMO);

    if (wave >= 4) {
        const int ptid = tid - 256;
        const int sx = __builtin_amdgcn_readfirstlane(mt * 2);
        const int wk = ptid >> 5, wq = ptid & 31;
        const int wv = ((k0 + wk) * ND + n0 + 4 * wq) * 4;
        const int ww_off = 2 * XB + wk * 256 + ((wq ^ ((wk & 3) | (((wk >> 2) & 1) << 4))) * 8);
        const int xiv = mt * 32 * XB + ptid * 16;
        const int xrow = ptid >> 2, xj = ptid & 3;
        const int xv = ((m0 + xrow) * KD + k0 + 4 * xj) * 4;
        const int fsw = (0x1320 >> (((xrow >> 2) & 3) * 4)) & 3;
        const int xw_off = xrow * 64 + ((xj ^ fsw) * 16);
        u32x4 r[DEPTH];
        unsigned call = 0;
        auto fl = [](unsigned v) { return __uint_as_float(v); };
        auto issue = [&](auto nv) {
            constexpr int n = decltype(nv)::value;
            if constexpr (n < SEQ_N) {
                constexpr Unit it = seq_unit(n);
                if constexpr (it.kind == 2 && it.st == FS && it.i == 0) {
                    const unsigned long long t0 = __builtin_amdgcn_s_memrealtime();
                    while (true) {
                        unsigned long long v = __hip_atomic_load((gu64*)xc + lane,
                                                                 __ATOMIC_RELAXED, __HIP_MEMORY_SCOPE_AGENT);
                        if (__all((unsigned)v == call && (unsigned)(v >> 32) == call)) break;
                        if (__builtin_amdgcn_s_memrealtime() - t0 > DEADLINE_TICKS) {
                            if (lane == 0) __hip_atomic_store(tmo, call, __ATOMIC_RELAXED, __HIP_MEMORY_SCOPE_AGENT);
                            break;
                        }
                        __builtin_amdgcn_s_sleep(1);
                    }
                    asm volatile("" ::: "memory");
                    SB0;
                }
                if constexpr (it.kind == 0) {
                    if constexpr (it.st & 2)
                        r[n % DEPTH] = __builtin_amdgcn_raw_buffer_load_b128(rw, wv, ((it.st ^ sx) * 32 + 8 * it.i) * ND * 4, 2);
                    else
                        r[n % DEPTH] = __builtin_amdgcn_raw_buffer_load_b128(rw, wv, ((it.st ^ sx) * 32 + 8 * it.i) * ND * 4, 0);
                } else if constexpr (it.kind == 1) {
                    r[n % DEPTH] = __builtin_amdgcn_raw_buffer_load_b128(rx, xv + (it.i & 1) * 64,
                                                                         (it.st ^ sx) * 128 + (it.i >> 1) * 64 * KD * 4, 0);
                } else {
                    r[n % DEPTH] = __builtin_amdgcn_raw_buffer_load_b128(ri, xiv, (it.st ^ sx) * XB + it.i * 4096, 0);
                }
            }
        };
        static_for<DEPTH>(issue);
        call = __hip_atomic_load(seqp, __ATOMIC_RELAXED, __HIP_MEMORY_SCOPE_AGENT) + 1;
        SB0;
        static_for<SEQ_N>([&](auto nv) {
            constexpr int n = decltype(nv)::value;
            constexpr Unit it = seq_unit(n);
            if constexpr (!(it.kind == 1 && !(it.i & 1))) {
                if constexpr (it.kind == 0) {
                    u32x2 w2;
                    w2[0] = cvt2(fl(r[n % DEPTH][0]), fl(r[n % DEPTH][1]));
                    w2[1] = cvt2(fl(r[n % DEPTH][2]), fl(r[n % DEPTH][3]));
                    *reinterpret_cast<u32x2*>(smem + ww_off + (it.st & 1) * WB + it.i * 8 * 256) = w2;
                } else if constexpr (it.kind == 1) {
                    constexpr int m = (n + DEPTH - 1) % DEPTH;
                    u32x4 v;
                    v[0] = cvt2(fl(r[m][0]), fl(r[m][1]));
                    v[1] = cvt2(fl(r[m][2]), fl(r[m][3]));
                    v[2] = cvt2(fl(r[n % DEPTH][0]), fl(r[n % DEPTH][1]));
                    v[3] = cvt2(fl(r[n % DEPTH][2]), fl(r[n % DEPTH][3]));
                    *reinterpret_cast<u32x4*>(smem + (it.st & 1) * XB + xw_off + (it.i >> 1) * 64 * 64) = v;
                } else {
                    *reinterpret_cast<u32x4*>(smem + (it.st & 1) * XB + ptid * 16 + it.i * 4096) = r[n % DEPTH];
                }
                SB0;
                if constexpr (it.kind == 1) issue(IC<n - 1 + DEPTH>{});
                issue(IC<n + DEPTH>{});
                SB0;
                if constexpr (it.last) {
                    asm volatile("s_waitcnt lgkmcnt(0)" ::: "memory");
                    __builtin_amdgcn_s_barrier();
                    asm volatile("" ::: "memory");
                    SB0;
                }
            }
        });
        if (wave < 7) {
            const int sp = wave - 4, src = (sl + 1 + sp) & 3;
            __amdgpu_buffer_rsrc_t rs = __builtin_amdgcn_make_buffer_rsrc((void*)(ws + WS_SCR), 0, (int)SCR_BYTES, 0x00020000);
            gu32* ef = (gu32*)(ws + WS_EFLG) + ((grp * 4 + sl) * 4 + (lane & 3)) * 4 + src;
            const unsigned long long t0 = __builtin_amdgcn_s_memrealtime();
            unsigned bad = 0;
            while (true) {
                if (__all(__hip_atomic_load(ef, __ATOMIC_RELAXED, __HIP_MEMORY_SCOPE_AGENT) == call)) break;
                if (__builtin_amdgcn_s_memrealtime() - t0 > DEADLINE_TICKS) {
                    if (lane == 0) __hip_atomic_store(tmo, call, __ATOMIC_RELAXED, __HIP_MEMORY_SCOPE_AGENT);
                    bad = 1;
                    break;
                }
                __builtin_amdgcn_s_sleep(1);
            }
            asm volatile("" ::: "memory");
            SB0;
#pragma unroll
            for (int e = 0; e < 16; ++e)
                __builtin_amdgcn_raw_ptr_buffer_load_lds(rs,
                    (__attribute__((address_space(3))) void*)(smem + EXO + ((e >> 2) * 12 + sp * 4 + (e & 3)) * 1024), 16,
                    lane * 16, (((grp * 4 + sl) * 3 + sp) * 16 + e) * 1024, 0, 16);
            if (lane == 0) sbc[12 + sp] = bad;
            asm volatile("s_waitcnt vmcnt(0) lgkmcnt(0)" ::: "memory");
            __builtin_amdgcn_s_barrier();
        }
        return;
    }

    const unsigned call = __hip_atomic_load(seqp, __ATOMIC_RELAXED, __HIP_MEMORY_SCOPE_AGENT) + 1;
    unsigned dummy = 0;
    {
        const int j = tid & 3, r16 = (tid >> 2) & 15;
        const int R = 16 * idx + r16, rt = R & 255;
        const int xs = (R * KD + k0 + wave * 32 + 4 * j) * 4;
        const int fsw = (0x1320 >> (((rt >> 2) & 3) * 4)) & 3;
        const int xd = (((R >> 8) * 32 + wave) * 256 + rt) * 64 + ((j ^ fsw) * 16);
        {
            const int tx = ((16 * idx + (lane & 15)) * KD + k0) * 4 + ((lane >> 4) + 4 * wave) * 128;
            if (4 * wave < FS) TOUCH(dummy, tx, rx, 0);
        }
        f32x4 a[8][2];
#pragma unroll
        for (int i = FS / 4; i < 8; ++i) { a[i][0] = ldb<0>(rx, xs, i * 512); a[i][1] = ldb<0>(rx, xs + 64, i * 512); }
#pragma unroll
        for (int i = FS / 4; i < 8; ++i) {
            u32x4 v;
            v[0] = cvt2(a[i][0][0], a[i][0][1]); v[1] = cvt2(a[i][0][2], a[i][0][3]);
            v[2] = cvt2(a[i][1][0], a[i][1][1]); v[3] = cvt2(a[i][1][2], a[i][1][3]);
            __builtin_amdgcn_raw_buffer_store_b128(v, ri, xd, i * 4 * XB, 16);
        }
    }
    const int wm = wave >> 1, wn = wave & 1;
    const int fr = (0x1320 >> (((c >> 2) & 3) * 4)) & 3;
    const int xr_off = (128 * wm + c) * 64 + ((h ^ fr) * 16);
    const int q4 = c >> 2, p4 = c & 3;
    int wr_off[4];
#pragma unroll
    for (int t = 0; t < 4; ++t)
        wr_off[t] = 2 * XB + (4 * h + q4) * 256 + ((((16 * wn) ^ ((h & 1) << 4)) + 4 * p4 + (t ^ q4)) * 8);

    f32x4 acc[4][8];
#pragma unroll
    for (int t = 0; t < 4; ++t)
#pragma unroll
        for (int u = 0; u < 8; ++u) acc[t][u] = f32x4{0.f, 0.f, 0.f, 0.f};
    bf16x8 xf[8], wf[2][4];
    auto rdw = [&](int st, int tt) {
        const char* wb = smem + (st & 1) * WB;
        bf16x4 lo = __builtin_amdgcn_ds_read_tr16_b64_v4bf16((__attribute__((address_space(3))) bf16x4*)(wb + wr_off[tt]));
        bf16x4 hi = __builtin_amdgcn_ds_read_tr16_b64_v4bf16((__attribute__((address_space(3))) bf16x4*)(wb + wr_off[tt] + 4096));
        return __builtin_shufflevector(lo, hi, 0, 1, 2, 3, 4, 5, 6, 7);
    };
    SB0;
    __syncthreads();
    SB0;
#pragma unroll
    for (int u = 0; u < 8; ++u) xf[u] = *reinterpret_cast<const bf16x8*>(smem + xr_off + u * 1024);
#pragma unroll
    for (int tt = 0; tt < 4; ++tt) wf[0][tt] = rdw(0, tt);
    SB0;
    __syncthreads();
    SB0;
    STAMP(1);
#pragma unroll
    for (int t = 0; t < NT; ++t) {
        const char* xb = smem + ((t + 1) & 1) * XB;
#pragma unroll
        for (int u = 0; u < 8; ++u) {
#pragma unroll
            for (int tt = 0; tt < 4; ++tt)
                acc[tt][u] = __builtin_amdgcn_mfma_f32_16x16x32_bf16(wf[t & 1][tt], xf[u], acc[tt][u], 0, 0, 0);
            if (t + 1 < NT) {
                xf[u] = *reinterpret_cast<const bf16x8*>(xb + xr_off + u * 1024);
                if (u < 4) wf[(t + 1) & 1][u] = rdw(t + 1, u);
            }
            if (t == 1 && u == 6) {
                asm volatile("s_waitcnt vmcnt(0)" :: "v"(dummy) : "memory");
                if (lane == 0) __hip_atomic_store(xc + idx * 4 + wave, call, __ATOMIC_RELAXED, __HIP_MEMORY_SCOPE_AGENT);
            }
            SB0;
        }
        if (t + 2 < NT) __syncthreads();
        SB0;
    }
    STAMP(2);

    __amdgpu_buffer_rsrc_t rs = __builtin_amdgcn_make_buffer_rsrc((void*)(ws + WS_SCR), 0, (int)SCR_BYTES, 0x00020000);
    f32x4 fin[2][4];
#pragma unroll
    for (int d = 0; d < 4; ++d) {
        if (d != sl) {
            const int sp = (sl - d - 1) & 3;
#pragma unroll
            for (int e = 0; e < 4; ++e) {
                const int u = 2 * d + (e >> 1), hf = e & 1;
                u32x4 v;
                v[0] = cvt2(acc[2 * hf][u][0], acc[2 * hf][u][1]);
                v[1] = cvt2(acc[2 * hf][u][2], acc[2 * hf][u][3]);
                v[2] = cvt2(acc[2 * hf + 1][u][0], acc[2 * hf + 1][u][1]);
                v[3] = cvt2(acc[2 * hf + 1][u][2], acc[2 * hf + 1][u][3]);
                __builtin_amdgcn_raw_buffer_store_b128(v, rs, lane * 16,
                    ((((grp * 4 + d) * 3 + sp) * 4 + wave) * 4 + e) * 1024, 16);
            }
        } else {
#pragma unroll
            for (int j = 0; j < 2; ++j)
#pragma unroll
                for (int tt = 0; tt < 4; ++tt) { fin[j][tt] = acc[tt][2 * d + j]; asm volatile("" : "+v"(fin[j][tt])); }
        }
    }
    if (tid == 0) __hip_atomic_store(seqp, call, __ATOMIC_RELAXED, __HIP_MEMORY_SCOPE_AGENT);
    asm volatile("s_waitcnt vmcnt(0)" ::: "memory");
    const unsigned long long t0 = __builtin_amdgcn_s_memrealtime();
    bool fail = false;
    {
        gu32* ef = (gu32*)(ws + WS_EFLG);
        if (lane < 4 && lane != sl)
            __hip_atomic_store(ef + ((grp * 4 + lane) * 4 + wave) * 4 + sl, call, __ATOMIC_RELAXED, __HIP_MEMORY_SCOPE_AGENT);
        STAMP(3);
    }
    LDS_BARRIER();
    fail = (sbc[12] | sbc[13] | sbc[14]) != 0;
    STAMP(4);
    {
        unsigned sel_lo = 0x00003f80u, sel_hi = 0x3f800000u;
        asm volatile("" : "+s"(sel_lo), "+s"(sel_hi));
        u32x4 pv[3][4];
#pragma unroll
        for (int i = 0; i < 3; ++i)
#pragma unroll
            for (int e = 0; e < 4; ++e)
                pv[i][e] = *reinterpret_cast<const u32x4*>(smem + EXO + wave * 12288 + lane * 16 + (i * 4 + e) * 1024);
#pragma unroll
        for (int i = 0; i < 3; ++i)
#pragma unroll
            for (int j = 0; j < 2; ++j)
#pragma unroll
                for (int hf = 0; hf < 2; ++hf)
#pragma unroll
                    for (int e = 0; e < 4; ++e) {
                        const unsigned pw = pv[i][2 * j + hf][e];
                        const bf16x2 p2 = __builtin_bit_cast(bf16x2, pw);
                        fin[j][2 * hf + (e >> 1)][(e & 1) * 2] = __builtin_amdgcn_fdot2_f32_bf16(
                            p2, __builtin_bit_cast(bf16x2, sel_lo), fin[j][2 * hf + (e >> 1)][(e & 1) * 2], false);
                        fin[j][2 * hf + (e >> 1)][(e & 1) * 2 + 1] = __builtin_amdgcn_fdot2_f32_bf16(
                            p2, __builtin_bit_cast(bf16x2, sel_hi), fin[j][2 * hf + (e >> 1)][(e & 1) * 2 + 1], false);
                    }
    }

    float am = 0.f;
#pragma unroll
    for (int j = 0; j < 2; ++j)
#pragma unroll
        for (int tt = 0; tt < 4; ++tt)
#pragma unroll
            for (int r = 0; r < 4; ++r) am = fmaxf(am, fabsf(fin[j][tt][r]));
    {
        const unsigned wm_ = wave_umax(__float_as_uint(am));
        if (lane == 0) sbc[wave] = wm_;
    }
    LDS_BARRIER();
    STAMP(5);
    const size_t par = (size_t)(call & 1u) * 16384;
    if (wave == 0 && lane < NR) {
        unsigned a = max(max(sbc[0], sbc[1]), max(sbc[2], sbc[3]));
        gchar* pws = (gchar*)ws_all[lane];
        __hip_atomic_store((gu64*)(pws + WS_AMAX + par + (rank * NWG + b) * 8),
                           ((unsigned long long)call << 32) | a, __ATOMIC_RELAXED, __HIP_MEMORY_SCOPE_SYSTEM);
    }
    STAMP(6);
    __amdgpu_buffer_rsrc_t rg = __builtin_amdgcn_make_buffer_rsrc((void*)(ws + WS_GMAX), 0, 2048, 0x00020000);
    if (b < 8) {
        __amdgpu_buffer_rsrc_t ra = __builtin_amdgcn_make_buffer_rsrc((void*)(ws + WS_AMAX + par), 0, 16384, 0x00020000);
        unsigned ga = 0, pend = 15;
        while (true) {
            asm volatile("" ::: "memory");
            u32x4 v[4];
#pragma unroll
            for (int i = 0; i < 4; ++i)
                if (pend & (1u << i)) v[i] = __builtin_amdgcn_raw_buffer_load_b128(ra, lane * 16, wave * 4096 + i * 1024, 17);
#pragma unroll
            for (int i = 0; i < 4; ++i)
                if ((pend & (1u << i)) && v[i][1] == call && v[i][3] == call) {
                    ga = max(ga, max(v[i][0], v[i][2]));
                    pend &= ~(1u << i);
                }
            if (__all(pend == 0)) break;
            if (__builtin_amdgcn_s_memrealtime() - t0 > DEADLINE_TICKS) {
                if (lane == 0) __hip_atomic_store(tmo, call, __ATOMIC_RELAXED, __HIP_MEMORY_SCOPE_AGENT);
                fail = true;
                break;
            }
        }
        ga = wave_umax(ga);
        if (lane == 0) { sbc[4 + wave] = ga; sbc[8 + wave] = fail ? 1u : 0u; }
        LDS_BARRIER();
        if (tid == 0 && (sbc[8] | sbc[9] | sbc[10] | sbc[11]) == 0) {
            u32x2 g2; g2[0] = max(max(sbc[4], sbc[5]), max(sbc[6], sbc[7])); g2[1] = call;
            __builtin_amdgcn_raw_buffer_store_b64(g2, rg, b * 256, 0, 16);
        }
    } else {
        if (wave == 0) {
            unsigned ga = 0;
            while (true) {
                asm volatile("" ::: "memory");
                u32x2 v = __builtin_amdgcn_raw_buffer_load_b64(rg, xcd * 256, 0, 16);
                ga = v[0];
                if (__builtin_amdgcn_readfirstlane(v[1]) == call) break;
                if (__builtin_amdgcn_s_memrealtime() - t0 > DEADLINE_TICKS) {
                    if (lane == 0) __hip_atomic_store(tmo, call, __ATOMIC_RELAXED, __HIP_MEMORY_SCOPE_AGENT);
                    fail = true;
                    break;
                }
                __builtin_amdgcn_s_sleep(AMAX_SLEEP);
            }
            if (lane == 0) { sbc[4] = ga; sbc[5] = 0; sbc[6] = 0; sbc[7] = 0; }
        }
        if (lane == 0) sbc[8 + wave] = fail ? 1u : 0u;
        LDS_BARRIER();
    }
    STAMP(7);
    const float gmax = __uint_as_float(max(max(sbc[4], sbc[5]), max(sbc[6], sbc[7])));
    const bool good = (sbc[8] | sbc[9] | sbc[10] | sbc[11]) == 0;

    const float inv = gmax > 0.f ? 448.0f / gmax : 0.f;
    const unsigned dst = (unsigned)(n0 / NS);
    if (dst < (unsigned)NR) {
        __amdgpu_buffer_rsrc_t ro = __builtin_amdgcn_make_buffer_rsrc(out_all[dst], 0, NR * MR * NS, 0x00020000);
#pragma unroll
        for (int j = 0; j < 2; ++j) {
            u32x4 q;
#pragma unroll
            for (int tt = 0; tt < 4; ++tt) {
                float v0 = __builtin_amdgcn_fmed3f(fin[j][tt][0] * inv, -448.f, 448.f);
                float v1 = __builtin_amdgcn_fmed3f(fin[j][tt][1] * inv, -448.f, 448.f);
                float v2 = __builtin_amdgcn_fmed3f(fin[j][tt][2] * inv, -448.f, 448.f);
                float v3 = __builtin_amdgcn_fmed3f(fin[j][tt][3] * inv, -448.f, 448.f);
                q[tt] = good ? pack_fp8x4(v0, v1, v2, v3) : 0x7f7f7f7fu;
            }
            __builtin_amdgcn_raw_buffer_store_b128(q, ro,
                (rank * MR + m0 + 128 * wm + 32 * sl + 16 * j + c) * NS + (n0 % NS) + 64 * wn + 16 * h, 0, 17);
        }
        asm volatile("s_waitcnt vmcnt(0)" ::: "memory");
        LDS_BARRIER();
        STAMP(8);
        if (tid == 0) {
            gchar* pws = (gchar*)ws_all[dst];
            __hip_atomic_store((gu32*)(pws + WS_DONE + (rank * 32 + ((nt & 3) * 2 + mt) * 4 + sl) * 4),
                               call, __ATOMIC_RELAXED, __HIP_MEMORY_SCOPE_SYSTEM);
        }
    }
    if (b == WAIT_WG && wave == 0) {
        STAMP(10);
        __amdgpu_buffer_rsrc_t rd = __builtin_amdgcn_make_buffer_rsrc((void*)(ws + WS_DONE), 0, 1024, 0x00020000);
        const unsigned long long t1 = __builtin_amdgcn_s_memrealtime();
        while (true) {
            asm volatile("" ::: "memory");
            u32x4 v = __builtin_amdgcn_raw_buffer_load_b128(rd, lane * 16, 0, 17);
            bool ok = v[0] >= call && v[1] >= call && v[2] >= call && v[3] >= call;
            if (__all(ok)) break;
            if (__builtin_amdgcn_s_memrealtime() - t1 > DEADLINE_TICKS) {
                if (lane == 0) {
                    __hip_atomic_store(tmo, call, __ATOMIC_RELAXED, __HIP_MEMORY_SCOPE_AGENT);
                    ((gchar*)out_all[rank])[(size_t)rank * MR * NS] = 0x7f;
                }
                break;
            }
        }
        __builtin_amdgcn_fence(__ATOMIC_ACQUIRE, "");
        STAMP(11);
    }
}

}

extern "C" void kernel_launch(int rank, int nranks, ncclComm_t comm,
                              hipStream_t stream,
                              void* const* d_in, const int* in_sizes, int n_in,
                              void* d_out, int out_size,
                              const int* in_dtypes, int out_dtype,
                              void* const* d_out_all,
                              void* const* d_ws_all, size_t ws_bytes) {
    (void)comm; (void)in_sizes; (void)n_in; (void)d_out; (void)out_size; (void)in_dtypes; (void)out_dtype;
    if (nranks != NR || ws_bytes < WS_END) {
        fprintf(stderr, "unsupported configuration (ws_bytes %zu)\n", ws_bytes);
        return;
    }
    hipLaunchKernelGGL(fused_gemm_a2a, dim3(NWG), dim3(NTHR), 0, stream,
                       (const float*)d_in[0], (const float*)d_in[1], d_out_all, d_ws_all, rank);
    HIP_CHECK(hipGetLastError());
}
```

```cpp
#include <hip/hip_runtime.h>
#include <rccl/rccl.h>
#include <cstdio>
#include <cstdint>
#include <utility>

#define HIP_CHECK(expr) do { hipError_t _e = (expr); if (_e != hipSuccess) { \
    fprintf(stderr, "HIP error %s at %s:%d\n", hipGetErrorString(_e), __FILE__, __LINE__); \
    return; } } while (0)

namespace {

typedef __attribute__((ext_vector_type(4))) float f32x4;
typedef __attribute__((ext_vector_type(4))) unsigned u32x4;
typedef __attribute__((ext_vector_type(2))) unsigned u32x2;
typedef __attribute__((ext_vector_type(8))) __bf16 bf16x8;
typedef __attribute__((ext_vector_type(4))) __bf16 bf16x4;
typedef __attribute__((ext_vector_type(2))) __bf16 bf16x2;
#define GAS __attribute__((address_space(1)))
typedef GAS char gchar;
typedef GAS unsigned gu32;
typedef GAS unsigned long long gu64;

constexpr int NR = 8;
constexpr int MR = 512;
constexpr int KD = 4096;
constexpr int ND = 4096;
constexpr int NS = 512;
constexpr int BM = 256, BN = 128, BK = 32, SK = 4, KS = KD / SK, NT = KS / BK;
constexpr int NWG = (MR / BM) * (ND / BN) * SK;
constexpr int NTHR = 512;
constexpr int WAIT_WG = 0;
#define AMAX_SLEEP 2
constexpr int FS = 8;
constexpr int DEPTH = 12;
constexpr int XB = BM * 64, WB = BK * 256;
constexpr int EXO = 2 * (XB + WB) + 64, EXB = 4 * 12 * 1024;
constexpr unsigned long long DEADLINE_TICKS = 50000000ull;

constexpr size_t WS_TMO   = 384;
constexpr size_t WS_DONE  = 2048;
constexpr size_t WS_XFLG  = 16384;
constexpr size_t WS_SEQ   = 32768;
constexpr size_t WS_EFLG  = 40960;
constexpr size_t WS_GMAX  = 57344;
constexpr size_t WS_AMAX  = 65536;
constexpr size_t WS_SCR   = 1 << 20;
constexpr size_t SCR_BYTES = (size_t)64 * 4 * 3 * 4 * 4096;
constexpr size_t WS_IMG   = WS_SCR + SCR_BYTES;
constexpr size_t WS_END   = WS_IMG + ((size_t)8 << 20);

__device__ __forceinline__ unsigned cvt2(float a, float b) {
    bf16x2 r; r[0] = (__bf16)a; r[1] = (__bf16)b; return __builtin_bit_cast(unsigned, r);
}
template <int AUX>
__device__ __forceinline__ f32x4 ldb(__amdgpu_buffer_rsrc_t r, int voff, int soff) {
    return __builtin_bit_cast(f32x4, __builtin_amdgcn_raw_buffer_load_b128(r, voff, soff, AUX));
}
__device__ __forceinline__ unsigned pack_fp8x4(float a, float b, float c, float d) {
    int r = 0;
    r = __builtin_amdgcn_cvt_pk_fp8_f32(a, b, r, false);
    r = __builtin_amdgcn_cvt_pk_fp8_f32(c, d, r, true);
    return (unsigned)r;
}
__device__ __forceinline__ unsigned wave_umax(unsigned m) {
    m = max(m, (unsigned)__builtin_amdgcn_update_dpp(0, (int)m, 0x111, 0xf, 0xf, true));
    m = max(m, (unsigned)__builtin_amdgcn_update_dpp(0, (int)m, 0x112, 0xf, 0xf, true));
    m = max(m, (unsigned)__builtin_amdgcn_update_dpp(0, (int)m, 0x114, 0xf, 0xf, true));
    m = max(m, (unsigned)__builtin_amdgcn_update_dpp(0, (int)m, 0x118, 0xf, 0xf, true));
    m = max(m, (unsigned)__builtin_amdgcn_update_dpp(0, (int)m, 0x142, 0xa, 0xf, true));
    m = max(m, (unsigned)__builtin_amdgcn_update_dpp(0, (int)m, 0x143, 0xc, 0xf, true));
    return (unsigned)__builtin_amdgcn_readlane((int)m, 63);
}
template <int V> struct IC { static constexpr int value = V; };
template <class Fn, int... I>
__device__ __forceinline__ void static_for_impl(Fn&& f, std::integer_sequence<int, I...>) { (f(IC<I>{}), ...); }
template <int N, class Fn>
__device__ __forceinline__ void static_for(Fn&& f) { static_for_impl(f, std::make_integer_sequence<int, N>{}); }
struct Unit { int kind, st, i; bool last; };
constexpr int SEQ_N = FS * 12 + (NT - FS) * 8;
constexpr Unit seq_unit(int n) {
    if (n < FS * 12) { const int st = n / 12, j = n % 12; return j < 4 ? Unit{0, st, j, false} : Unit{1, st, j - 4, j == 11}; }
    const int st = FS + (n - FS * 12) / 8, j = (n - FS * 12) % 8;
    return j < 4 ? Unit{0, st, j, false} : Unit{2, st, j - 4, j == 7};
}
#define SB0 __builtin_amdgcn_sched_barrier(0)
#define LDS_BARRIER() asm volatile("s_waitcnt lgkmcnt(0)\n\ts_barrier" ::: "memory")
#define TOUCH(dummy, voff, rsrc, soff) \
    asm volatile("buffer_load_dword %0, %1, %2, %3 offen" : "+v"(dummy) : "v"(voff), "s"(rsrc), "s"(soff))
#define STAMP(i) do { } while (0)

__global__ void __launch_bounds__(NTHR, 2)
fused_gemm_a2a(const float* __restrict__ X, const float* __restrict__ W,
               void* const* __restrict__ out_all, void* const* __restrict__ ws_all, int rank) {
    __shared__ __attribute__((aligned(16))) char smem[2 * (XB + WB) + 64 + EXB];
    unsigned* sbc = reinterpret_cast<unsigned*>(smem + 2 * (XB + WB));

    const int tid = threadIdx.x, lane = tid & 63;
    const int wave = __builtin_amdgcn_readfirstlane(tid >> 6);
    const int c = lane & 15, h = lane >> 4;
    const int b = blockIdx.x, xcd = b & 7, idx = b >> 3;
    const int sl = xcd & 3, mt = idx & 1, nt = (xcd >> 2) * 16 + (idx >> 1);
    const int m0 = mt * BM, n0 = nt * BN, k0 = sl * KS;
    const int grp = mt * 32 + nt;

    gchar* ws = (gchar*)ws_all[rank];
    STAMP(0);
    __amdgpu_buffer_rsrc_t rx = __builtin_amdgcn_make_buffer_rsrc((void*)X, 0, MR * KD * 4, 0x00020000);
    __amdgpu_buffer_rsrc_t rw = __builtin_amdgcn_make_buffer_rsrc((void*)W, 0, KD * ND * 4, 0x00020000);
    __amdgpu_buffer_rsrc_t ri = __builtin_amdgcn_make_buffer_rsrc((void*)(ws + WS_IMG + ((size_t)xcd << 20)), 0, 1 << 20, 0x00020000);
    gu32* seqp = (gu32*)(ws + WS_SEQ + b * 4);
    gu32* xc = (gu32*)(ws + WS_XFLG + xcd * 512);
    gu32* tmo = (gu32*)(ws + WS_TMO);

    if (wave >= 4) {
        const int ptid = tid - 256;
        const int sx = __builtin_amdgcn_readfirstlane(mt * 2);
        const int wk = ptid >> 5, wq = ptid & 31;
        const int wv = ((k0 + wk) * ND + n0 + 4 * wq) * 4;
        const int ww_off = 2 * XB + wk * 256 + ((wq ^ ((wk & 3) | (((wk >> 2) & 1) << 4))) * 8);
        const int xiv = mt * 32 * XB + ptid * 16;
        const int xrow = ptid >> 2, xj = ptid & 3;
        const int xv = ((m0 + xrow) * KD + k0 + 4 * xj) * 4;
        const int fsw = (0x1320 >> (((xrow >> 2) & 3) * 4)) & 3;
        const int xw_off = xrow * 64 + ((xj ^ fsw) * 16);
        u32x4 r[DEPTH];
        unsigned call = 0;
        auto fl = [](unsigned v) { return __uint_as_float(v); };
        auto issue = [&](auto nv) {
            constexpr int n = decltype(nv)::value;
            if constexpr (n < SEQ_N) {
                constexpr Unit it = seq_unit(n);
                if constexpr (it.kind == 2 && it.st == FS && it.i == 0) {
                    const unsigned long long t0 = __builtin_amdgcn_s_memrealtime();
                    while (true) {
                        unsigned long long v = __hip_atomic_load((gu64*)xc + lane,
                                                                 __ATOMIC_RELAXED, __HIP_MEMORY_SCOPE_AGENT);
                        if (__all((unsigned)v == call && (unsigned)(v >> 32) == call)) break;
                        if (__builtin_amdgcn_s_memrealtime() - t0 > DEADLINE_TICKS) {
                            if (lane == 0) __hip_atomic_store(tmo, call, __ATOMIC_RELAXED, __HIP_MEMORY_SCOPE_AGENT);
                            break;
                        }
                        __builtin_amdgcn_s_sleep(1);
                    }
                    asm volatile("" ::: "memory");
                    SB0;
                }
                if constexpr (it.kind == 0) {
                    r[n % DEPTH] = __builtin_amdgcn_raw_buffer_load_b128(rw, wv, ((it.st ^ sx) * 32 + 8 * it.i) * ND * 4, 0);
                } else if constexpr (it.kind == 1) {
                    r[n % DEPTH] = __builtin_amdgcn_raw_buffer_load_b128(rx, xv + (it.i & 1) * 64,
                                                                         (it.st ^ sx) * 128 + (it.i >> 1) * 64 * KD * 4, 0);
                } else {
                    r[n % DEPTH] = __builtin_amdgcn_raw_buffer_load_b128(ri, xiv, (it.st ^ sx) * XB + it.i * 4096, 0);
                }
            }
        };
        static_for<DEPTH>(issue);
        call = __hip_atomic_load(seqp, __ATOMIC_RELAXED, __HIP_MEMORY_SCOPE_AGENT) + 1;
        SB0;
        static_for<SEQ_N>([&](auto nv) {
            constexpr int n = decltype(nv)::value;
            constexpr Unit it = seq_unit(n);
            if constexpr (!(it.kind == 1 && !(it.i & 1))) {
                if constexpr (it.kind == 0) {
                    u32x2 w2;
                    w2[0] = cvt2(fl(r[n % DEPTH][0]), fl(r[n % DEPTH][1]));
                    w2[1] = cvt2(fl(r[n % DEPTH][2]), fl(r[n % DEPTH][3]));
                    *reinterpret_cast<u32x2*>(smem + ww_off + (it.st & 1) * WB + it.i * 8 * 256) = w2;
                } else if constexpr (it.kind == 1) {
                    constexpr int m = (n + DEPTH - 1) % DEPTH;
                    u32x4 v;
                    v[0] = cvt2(fl(r[m][0]), fl(r[m][1]));
                    v[1] = cvt2(fl(r[m][2]), fl(r[m][3]));
                    v[2] = cvt2(fl(r[n % DEPTH][0]), fl(r[n % DEPTH][1]));
                    v[3] = cvt2(fl(r[n % DEPTH][2]), fl(r[n % DEPTH][3]));
                    *reinterpret_cast<u32x4*>(smem + (it.st & 1) * XB + xw_off + (it.i >> 1) * 64 * 64) = v;
                } else {
                    *reinterpret_cast<u32x4*>(smem + (it.st & 1) * XB + ptid * 16 + it.i * 4096) = r[n % DEPTH];
                }
                SB0;
                if constexpr (it.kind == 1) issue(IC<n - 1 + DEPTH>{});
                issue(IC<n + DEPTH>{});
                SB0;
                if constexpr (it.last) {
                    asm volatile("s_waitcnt lgkmcnt(0)" ::: "memory");
                    __builtin_amdgcn_s_barrier();
                    asm volatile("" ::: "memory");
                    SB0;
                }
            }
        });
        if (wave < 7) {
            const int sp = wave - 4, src = (sl + 1 + sp) & 3;
            __amdgpu_buffer_rsrc_t rs = __builtin_amdgcn_make_buffer_rsrc((void*)(ws + WS_SCR), 0, (int)SCR_BYTES, 0x00020000);
            gu32* ef = (gu32*)(ws + WS_EFLG) + ((grp * 4 + sl) * 4 + (lane & 3)) * 4 + src;
            const unsigned long long t0 = __builtin_amdgcn_s_memrealtime();
            unsigned bad = 0;
            while (true) {
                if (__all(__hip_atomic_load(ef, __ATOMIC_RELAXED, __HIP_MEMORY_SCOPE_AGENT) == call)) break;
                if (__builtin_amdgcn_s_memrealtime() - t0 > DEADLINE_TICKS) {
                    if (lane == 0) __hip_atomic_store(tmo, call, __ATOMIC_RELAXED, __HIP_MEMORY_SCOPE_AGENT);
                    bad = 1;
                    break;
                }
                __builtin_amdgcn_s_sleep(1);
            }
            asm volatile("" ::: "memory");
            SB0;
#pragma unroll
            for (int e = 0; e < 16; ++e)
                __builtin_amdgcn_raw_ptr_buffer_load_lds(rs,
                    (__attribute__((address_space(3))) void*)(smem + EXO + ((e >> 2) * 12 + sp * 4 + (e & 3)) * 1024), 16,
                    lane * 16, (((grp * 4 + sl) * 3 + sp) * 16 + e) * 1024, 0, 16);
            if (lane == 0) sbc[12 + sp] = bad;
            asm volatile("s_waitcnt vmcnt(0) lgkmcnt(0)" ::: "memory");
            __builtin_amdgcn_s_barrier();
        }
        return;
    }

    const unsigned call = __hip_atomic_load(seqp, __ATOMIC_RELAXED, __HIP_MEMORY_SCOPE_AGENT) + 1;
    unsigned dummy = 0;
    {
        const int j = tid & 3, r16 = (tid >> 2) & 15;
        const int R = 16 * idx + r16, rt = R & 255;
        const int xs = (R * KD + k0 + wave * 32 + 4 * j) * 4;
        const int fsw = (0x1320 >> (((rt >> 2) & 3) * 4)) & 3;
        const int xd = (((R >> 8) * 32 + wave) * 256 + rt) * 64 + ((j ^ fsw) * 16);
        {
            const int tx = ((16 * idx + (lane & 15)) * KD + k0) * 4 + ((lane >> 4) + 4 * wave) * 128;
            if (4 * wave < FS) TOUCH(dummy, tx, rx, 0);
        }
        f32x4 a[8][2];
#pragma unroll
        for (int i = FS / 4; i < 8; ++i) { a[i][0] = ldb<0>(rx, xs, i * 512); a[i][1] = ldb<0>(rx, xs + 64, i * 512); }
#pragma unroll
        for (int i = FS / 4; i < 8; ++i) {
            u32x4 v;
            v[0] = cvt2(a[i][0][0], a[i][0][1]); v[1] = cvt2(a[i][0][2], a[i][0][3]);
            v[2] = cvt2(a[i][1][0], a[i][1][1]); v[3] = cvt2(a[i][1][2], a[i][1][3]);
            __builtin_amdgcn_raw_buffer_store_b128(v, ri, xd, i * 4 * XB, 16);
        }
    }
    const int wm = wave >> 1, wn = wave & 1;
    const int fr = (0x1320 >> (((c >> 2) & 3) * 4)) & 3;
    const int xr_off = (128 * wm + c) * 64 + ((h ^ fr) * 16);
    const int q4 = c >> 2, p4 = c & 3;
    int wr_off[4];
#pragma unroll
    for (int t = 0; t < 4; ++t)
        wr_off[t] = 2 * XB + (4 * h + q4) * 256 + ((((16 * wn) ^ ((h & 1) << 4)) + 4 * p4 + (t ^ q4)) * 8);

    f32x4 acc[4][8];
#pragma unroll
    for (int t = 0; t < 4; ++t)
#pragma unroll
        for (int u = 0; u < 8; ++u) acc[t][u] = f32x4{0.f, 0.f, 0.f, 0.f};
    bf16x8 xf[8], wf[2][4];
    auto rdw = [&](int st, int tt) {
        const char* wb = smem + (st & 1) * WB;
        bf16x4 lo = __builtin_amdgcn_ds_read_tr16_b64_v4bf16((__attribute__((address_space(3))) bf16x4*)(wb + wr_off[tt]));
        bf16x4 hi = __builtin_amdgcn_ds_read_tr16_b64_v4bf16((__attribute__((address_space(3))) bf16x4*)(wb + wr_off[tt] + 4096));
        return __builtin_shufflevector(lo, hi, 0, 1, 2, 3, 4, 5, 6, 7);
    };
    SB0;
    __syncthreads();
    SB0;
#pragma unroll
    for (int u = 0; u < 8; ++u) xf[u] = *reinterpret_cast<const bf16x8*>(smem + xr_off + u * 1024);
#pragma unroll
    for (int tt = 0; tt < 4; ++tt) wf[0][tt] = rdw(0, tt);
    SB0;
    __syncthreads();
    SB0;
    STAMP(1);
#pragma unroll
    for (int t = 0; t < NT; ++t) {
        const char* xb = smem + ((t + 1) & 1) * XB;
#pragma unroll
        for (int u = 0; u < 8; ++u) {
#pragma unroll
            for (int tt = 0; tt < 4; ++tt)
                acc[tt][u] = __builtin_amdgcn_mfma_f32_16x16x32_bf16(wf[t & 1][tt], xf[u], acc[tt][u], 0, 0, 0);
            if (t + 1 < NT) {
                xf[u] = *reinterpret_cast<const bf16x8*>(xb + xr_off + u * 1024);
                if (u < 4) wf[(t + 1) & 1][u] = rdw(t + 1, u);
            }
            if (t == 1 && u == 6) {
                asm volatile("s_waitcnt vmcnt(0)" :: "v"(dummy) : "memory");
                if (lane == 0) __hip_atomic_store(xc + idx * 4 + wave, call, __ATOMIC_RELAXED, __HIP_MEMORY_SCOPE_AGENT);
            }
            SB0;
        }
        if (t + 2 < NT) __syncthreads();
        SB0;
    }
    STAMP(2);

    __amdgpu_buffer_rsrc_t rs = __builtin_amdgcn_make_buffer_rsrc((void*)(ws + WS_SCR), 0, (int)SCR_BYTES, 0x00020000);
    f32x4 fin[2][4];
#pragma unroll
    for (int d = 0; d < 4; ++d) {
        if (d != sl) {
            const int sp = (sl - d - 1) & 3;
#pragma unroll
            for (int e = 0; e < 4; ++e) {
                const int u = 2 * d + (e >> 1), hf = e & 1;
                u32x4 v;
                v[0] = cvt2(acc[2 * hf][u][0], acc[2 * hf][u][1]);
                v[1] = cvt2(acc[2 * hf][u][2], acc[2 * hf][u][3]);
                v[2] = cvt2(acc[2 * hf + 1][u][0], acc[2 * hf + 1][u][1]);
                v[3] = cvt2(acc[2 * hf + 1][u][2], acc[2 * hf + 1][u][3]);
                __builtin_amdgcn_raw_buffer_store_b128(v, rs, lane * 16,
                    ((((grp * 4 + d) * 3 + sp) * 4 + wave) * 4 + e) * 1024, 16);
            }
        } else {
#pragma unroll
            for (int j = 0; j < 2; ++j)
#pragma unroll
                for (int tt = 0; tt < 4; ++tt) { fin[j][tt] = acc[tt][2 * d + j]; asm volatile("" : "+v"(fin[j][tt])); }
        }
    }
    if (tid == 0) __hip_atomic_store(seqp, call, __ATOMIC_RELAXED, __HIP_MEMORY_SCOPE_AGENT);
    asm volatile("s_waitcnt vmcnt(0)" ::: "memory");
    const unsigned long long t0 = __builtin_amdgcn_s_memrealtime();
    bool fail = false;
    {
        gu32* ef = (gu32*)(ws + WS_EFLG);
        if (lane < 4 && lane != sl)
            __hip_atomic_store(ef + ((grp * 4 + lane) * 4 + wave) * 4 + sl, call, __ATOMIC_RELAXED, __HIP_MEMORY_SCOPE_AGENT);
        STAMP(3);
    }
    LDS_BARRIER();
    fail = (sbc[12] | sbc[13] | sbc[14]) != 0;
    STAMP(4);
    {
        unsigned sel_lo = 0x00003f80u, sel_hi = 0x3f800000u;
        asm volatile("" : "+s"(sel_lo), "+s"(sel_hi));
        u32x4 pv[3][4];
#pragma unroll
        for (int i = 0; i < 3; ++i)
#pragma unroll
            for (int e = 0; e < 4; ++e)
                pv[i][e] = *reinterpret_cast<const u32x4*>(smem + EXO + wave * 12288 + lane * 16 + (i * 4 + e) * 1024);
#pragma unroll
        for (int i = 0; i < 3; ++i)
#pragma unroll
            for (int j = 0; j < 2; ++j)
#pragma unroll
                for (int hf = 0; hf < 2; ++hf)
#pragma unroll
                    for (int e = 0; e < 4; ++e) {
                        const unsigned pw = pv[i][2 * j + hf][e];
                        const bf16x2 p2 = __builtin_bit_cast(bf16x2, pw);
                        fin[j][2 * hf + (e >> 1)][(e & 1) * 2] = __builtin_amdgcn_fdot2_f32_bf16(
                            p2, __builtin_bit_cast(bf16x2, sel_lo), fin[j][2 * hf + (e >> 1)][(e & 1) * 2], false);
                        fin[j][2 * hf + (e >> 1)][(e & 1) * 2 + 1] = __builtin_amdgcn_fdot2_f32_bf16(
                            p2, __builtin_bit_cast(bf16x2, sel_hi), fin[j][2 * hf + (e >> 1)][(e & 1) * 2 + 1], false);
                    }
    }

    float am = 0.f;
#pragma unroll
    for (int j = 0; j < 2; ++j)
#pragma unroll
        for (int tt = 0; tt < 4; ++tt)
#pragma unroll
            for (int r = 0; r < 4; ++r) am = fmaxf(am, fabsf(fin[j][tt][r]));
    {
        const unsigned wm_ = wave_umax(__float_as_uint(am));
        if (lane == 0) sbc[wave] = wm_;
    }
    LDS_BARRIER();
    STAMP(5);
    const size_t par = (size_t)(call & 1u) * 16384;
    if (wave == 0 && lane < NR) {
        unsigned a = max(max(sbc[0], sbc[1]), max(sbc[2], sbc[3]));
        gchar* pws = (gchar*)ws_all[lane];
        __hip_atomic_store((gu64*)(pws + WS_AMAX + par + (rank * NWG + b) * 8),
                           ((unsigned long long)call << 32) | a, __ATOMIC_RELAXED, __HIP_MEMORY_SCOPE_SYSTEM);
    }
    STAMP(6);
    __amdgpu_buffer_rsrc_t rg = __builtin_amdgcn_make_buffer_rsrc((void*)(ws + WS_GMAX), 0, 2048, 0x00020000);
    if (b < 8) {
        __amdgpu_buffer_rsrc_t ra = __builtin_amdgcn_make_buffer_rsrc((void*)(ws + WS_AMAX + par), 0, 16384, 0x00020000);
        unsigned ga = 0, pend = 15;
        while (true) {
            asm volatile("" ::: "memory");
            u32x4 v[4];
#pragma unroll
            for (int i = 0; i < 4; ++i)
                if (pend & (1u << i)) v[i] = __builtin_amdgcn_raw_buffer_load_b128(ra, lane * 16, wave * 4096 + i * 1024, 17);
#pragma unroll
            for (int i = 0; i < 4; ++i)
                if ((pend & (1u << i)) && v[i][1] == call && v[i][3] == call) {
                    ga = max(ga, max(v[i][0], v[i][2]));
                    pend &= ~(1u << i);
                }
            if (__all(pend == 0)) break;
            if (__builtin_amdgcn_s_memrealtime() - t0 > DEADLINE_TICKS) {
                if (lane == 0) __hip_atomic_store(tmo, call, __ATOMIC_RELAXED, __HIP_MEMORY_SCOPE_AGENT);
                fail = true;
                break;
            }
        }
        ga = wave_umax(ga);
        if (lane == 0) { sbc[4 + wave] = ga; sbc[8 + wave] = fail ? 1u : 0u; }
        LDS_BARRIER();
        if (tid == 0 && (sbc[8] | sbc[9] | sbc[10] | sbc[11]) == 0) {
            u32x2 g2; g2[0] = max(max(sbc[4], sbc[5]), max(sbc[6], sbc[7])); g2[1] = call;
            __builtin_amdgcn_raw_buffer_store_b64(g2, rg, b * 256, 0, 16);
        }
    } else {
        if (wave == 0) {
            unsigned ga = 0;
            while (true) {
                asm volatile("" ::: "memory");
                u32x2 v = __builtin_amdgcn_raw_buffer_load_b64(rg, xcd * 256, 0, 16);
                ga = v[0];
                if (__builtin_amdgcn_readfirstlane(v[1]) == call) break;
                if (__builtin_amdgcn_s_memrealtime() - t0 > DEADLINE_TICKS) {
                    if (lane == 0) __hip_atomic_store(tmo, call, __ATOMIC_RELAXED, __HIP_MEMORY_SCOPE_AGENT);
                    fail = true;
                    break;
                }
                __builtin_amdgcn_s_sleep(AMAX_SLEEP);
            }
            if (lane == 0) { sbc[4] = ga; sbc[5] = 0; sbc[6] = 0; sbc[7] = 0; }
        }
        if (lane == 0) sbc[8 + wave] = fail ? 1u : 0u;
        LDS_BARRIER();
    }
    STAMP(7);
    const float gmax = __uint_as_float(max(max(sbc[4], sbc[5]), max(sbc[6], sbc[7])));
    const bool good = (sbc[8] | sbc[9] | sbc[10] | sbc[11]) == 0;

    const float inv = gmax > 0.f ? 448.0f / gmax : 0.f;
    const unsigned dst = (unsigned)(n0 / NS);
    if (dst < (unsigned)NR) {
        __amdgpu_buffer_rsrc_t ro = __builtin_amdgcn_make_buffer_rsrc(out_all[dst], 0, NR * MR * NS, 0x00020000);
#pragma unroll
        for (int j = 0; j < 2; ++j) {
            u32x4 q;
#pragma unroll
            for (int tt = 0; tt < 4; ++tt) {
                float v0 = __builtin_amdgcn_fmed3f(fin[j][tt][0] * inv, -448.f, 448.f);
                float v1 = __builtin_amdgcn_fmed3f(fin[j][tt][1] * inv, -448.f, 448.f);
                float v2 = __builtin_amdgcn_fmed3f(fin[j][tt][2] * inv, -448.f, 448.f);
                float v3 = __builtin_amdgcn_fmed3f(fin[j][tt][3] * inv, -448.f, 448.f);
                q[tt] = good ? pack_fp8x4(v0, v1, v2, v3) : 0x7f7f7f7fu;
            }
            __builtin_amdgcn_raw_buffer_store_b128(q, ro,
                (rank * MR + m0 + 128 * wm + 32 * sl + 16 * j + c) * NS + (n0 % NS) + 64 * wn + 16 * h, 0, 17);
        }
        asm volatile("s_waitcnt vmcnt(0)" ::: "memory");
        LDS_BARRIER();
        STAMP(8);
        if (tid == 0) {
            gchar* pws = (gchar*)ws_all[dst];
            __hip_atomic_store((gu32*)(pws + WS_DONE + (rank * 32 + ((nt & 3) * 2 + mt) * 4 + sl) * 4),
                               call, __ATOMIC_RELAXED, __HIP_MEMORY_SCOPE_SYSTEM);
        }
    }
    if (b == WAIT_WG && wave == 0) {
        STAMP(10);
        __amdgpu_buffer_rsrc_t rd = __builtin_amdgcn_make_buffer_rsrc((void*)(ws + WS_DONE), 0, 1024, 0x00020000);
        const unsigned long long t1 = __builtin_amdgcn_s_memrealtime();
        while (true) {
            asm volatile("" ::: "memory");
            u32x4 v = __builtin_amdgcn_raw_buffer_load_b128(rd, lane * 16, 0, 17);
            bool ok = v[0] >= call && v[1] >= call && v[2] >= call && v[3] >= call;
            if (__all(ok)) break;
            if (__builtin_amdgcn_s_memrealtime() - t1 > DEADLINE_TICKS) {
                if (lane == 0) {
                    __hip_atomic_store(tmo, call, __ATOMIC_RELAXED, __HIP_MEMORY_SCOPE_AGENT);
                    ((gchar*)out_all[rank])[(size_t)rank * MR * NS] = 0x7f;
                }
                break;
            }
        }
        __builtin_amdgcn_fence(__ATOMIC_ACQUIRE, "");
        STAMP(11);
    }
}

}

extern "C" void kernel_launch(int rank, int nranks, ncclComm_t comm,
                              hipStream_t stream,
                              void* const* d_in, const int* in_sizes, int n_in,
                              void* d_out, int out_size,
                              const int* in_dtypes, int out_dtype,
                              void* const* d_out_all,
                              void* const* d_ws_all, size_t ws_bytes) {
    (void)comm; (void)in_sizes; (void)n_in; (void)d_out; (void)out_size; (void)in_dtypes; (void)out_dtype;
    if (nranks != NR || ws_bytes < WS_END) {
        fprintf(stderr, "unsupported configuration (ws_bytes %zu)\n", ws_bytes);
        return;
    }
    hipLaunchKernelGGL(fused_gemm_a2a, dim3(NWG), dim3(NTHR), 0, stream,
                       (const float*)d_in[0], (const float*)d_in[1], d_out_all, d_ws_all, rank);
    HIP_CHECK(hipGetLastError());
}
```
